```python
import numpy as np

kernel_name = "gpu_a2a_gemm_m4096_k4096_n32768_bf16_silu"


ABSMAX_RTOL = 0.008

COMPUTE_DTYPE = "bfloat16"
EPILOGUE = "silu"
IN_QUANT = None
OUT_QUANT = None
IN_DTYPES = {"x": "bfloat16", "w_mat": "bfloat16"}
WORLD_SIZE = 8

def _to_bf16(x):
    u = np.ascontiguousarray(x, dtype=np.float32).view(np.uint32)
    return ((u + 0x7FFF + ((u >> 16) & 1)) & 0xFFFF0000).view(np.float32)

def setup_inputs():

    x = _to_bf16(np.random.randn(4096, 4096).astype(np.float32))
    w_mat = _to_bf16((np.random.randn(4096, 32768) / np.sqrt(4096)).astype(np.float32))
    return {"x": x, "w_mat": w_mat}

def _epilogue(y):
    return (y / (1.0 + np.exp(-np.clip(y, -60.0, 60.0)))).astype(np.float32)

def reference(x, w_mat):
    y = (x @ w_mat).astype(np.float32)
    return _epilogue(y)
```

```cpp
#include <hip/hip_runtime.h>
#include <rccl/rccl.h>
#include <cstdio>
#include <type_traits>

#define HIP_CHECK(expr) do { hipError_t _e = (expr); if (_e != hipSuccess) { \
    fprintf(stderr, "HIP error %s at %s:%d\n", hipGetErrorString(_e), __FILE__, __LINE__); \
    return; } } while (0)

namespace {

#define LAS __attribute__((address_space(3)))
typedef short bf16x8 __attribute__((ext_vector_type(8)));
typedef short s16x4 __attribute__((ext_vector_type(4)));
typedef float f32x4 __attribute__((ext_vector_type(4)));
typedef unsigned u32x4 __attribute__((ext_vector_type(4)));
typedef float f32x2 __attribute__((ext_vector_type(2)));
typedef __bf16 bf16x2_t __attribute__((ext_vector_type(2)));

constexpr int NRANK = 8, MR = 512, KS = 512, NN = 32768;
constexpr int NWG = 256;
constexpr unsigned WS_TMO = 0;
constexpr unsigned WS_CALLS = 65536;
constexpr unsigned WS_STAGE = 2u << 20, SLOT_BYTES = 8u << 20, FLAG_OFF = 32768;
constexpr size_t WS_NEED = (size_t)WS_STAGE + 2 * (size_t)SLOT_BYTES;
constexpr unsigned RING = 4, SLOTB = 32768;
constexpr unsigned LDS_XS = RING * SLOTB;
constexpr unsigned LDS_POLL = LDS_XS + 16384;
constexpr unsigned LDS_CTL = LDS_POLL + 1024;
constexpr unsigned LDS_BYTES = LDS_CTL + 16;
#define TS(i) ((void)0)
#define APOL_R " sc0 sc1"
#define LATEWAIT 1
#define SPOL " sc0 sc1 nt"
constexpr unsigned long long DEADLINE_TICKS = 100000000ull;

struct Params { const char* x; const char* w; char* out; char* ws[NRANK]; int rank; unsigned long long* stamps; };

#define WAIT_V(n) asm volatile("s_waitcnt vmcnt(%0)" :: "n"(n) : "memory")
#define WAIT_L0() do { asm volatile("" ::: "memory"); __builtin_amdgcn_s_waitcnt(0xC07F); asm volatile("" ::: "memory"); } while (0)
#define BAR() do { asm volatile("" ::: "memory"); __builtin_amdgcn_s_barrier(); asm volatile("" ::: "memory"); } while (0)
#define SCHED() __builtin_amdgcn_sched_barrier(0)
#define GLDS2P(POL, m0v, v0, v1, base) asm volatile("s_mov_b32 m0, %0\n\ts_nop 0\n\tglobal_load_lds_dwordx4 %1, %3" POL "\n\tglobal_load_lds_dwordx4 %2, %3 offset:1024" POL \
    :: "s"(m0v), "v"(v0), "v"(v1), "s"(base) : "memory")
#define GLDS2(m0v, v0, v1, base) GLDS2P("", m0v, v0, v1, base)

__device__ __forceinline__ unsigned pk_bf16(float lo, float hi) { f32x2 v = {lo, hi}; bf16x2_t b = __builtin_convertvector(v, bf16x2_t); return __builtin_bit_cast(unsigned, b); }
__device__ __forceinline__ float silu(float y) {
  const float c = __builtin_amdgcn_fmed3f(y, -60.f, 60.f);
  const float e = __builtin_amdgcn_exp2f(c * -1.44269504088896340736f);
  return y * __builtin_amdgcn_rcpf(1.0f + e);
}
__device__ __forceinline__ bool flags_ok(const u32x4 f, unsigned need) {
  const int m = min(min((int)(f[0] - need), (int)(f[1] - need)), min((int)(f[2] - need), (int)(f[3] - need)));
  return __builtin_amdgcn_ballot_w64(m < 0) == 0ull;
}

__global__ __launch_bounds__(512, 2) void a2a_gemm_silu(Params g) {
  __shared__ __attribute__((aligned(1024))) char smem[LDS_BYTES];
  LAS char* lds = (LAS char*)smem;
  const int tid = threadIdx.x, wid = __builtin_amdgcn_readfirstlane(tid >> 6), lane = tid & 63;
  const int wr = wid >> 2, wc = wid & 3, fr = lane & 15, fq = lane >> 4;
  const int bid = blockIdx.x, pn = (bid & 7) * 16 + (bid >> 4), pm = (bid >> 3) & 1;
  const unsigned rank = (unsigned)g.rank & 7u;
  char* const wsl = g.ws[rank];
  const unsigned ldsbase = (unsigned)(size_t)lds;

  unsigned voffA[2], voffB[2];
  const int obp = lane * 16, ob = obp ^ (((obp >> 9) & 1) << 5);
#pragma unroll
  for (int i = 0; i < 2; ++i) voffA[i] = (unsigned)((pm * 256 + (wid * 2 + i) * 16 + (ob >> 6)) * 1024 + ((ob >> 4) & 3) * 16 + 4096 - 1024 * i);
#pragma unroll
  for (int i = 0; i < 2; ++i) { const int kr = (wid * 2 + i) * 2 + (lane >> 5), f = (kr & 3) | (((kr >> 3) & 1) << 2), c = (lane & 31) ^ (f << 1);
    voffB[i] = (unsigned)(kr * 65536 + c * 16 + 4096 - 1024 * i); }
  const unsigned m0w = ldsbase + (unsigned)wid * 2048u;
  const char* const Wb = g.w + (size_t)pn * 512 - 4096;
  const char* ap = g.x + (size_t)rank * (MR * 1024) - 4096;
  const char* wp = Wb + ((size_t)(rank * 16u) << 21);
  unsigned wslot = 0;
  unsigned arow = 1024;
#define GLDS1P(POL, m0v, v0, base) asm volatile("s_mov_b32 m0, %0\n\ts_nop 0\n\tglobal_load_lds_dwordx4 %1, %2" POL :: "s"(m0v), "v"(v0), "s"(base) : "memory")
#define ISSUE(HELP, APOL) do { GLDS2P(APOL, m0w + wslot, voffA[0], voffA[1], ap); GLDS2(m0w + wslot + 16384u, voffB[0], voffB[1], wp); \
    if (HELP == 4) GLDS1P(APOL, m0w + wslot + 6144u, voffA[0], ap + 96u * arow); \
    if (HELP == 5) GLDS1P(APOL, m0w + wslot + 5120u, voffA[0], ap + 80u * arow); \
    if (HELP == 6) GLDS2(m0w + wslot + 2048u + 16384u, voffB[0], voffB[1], wp + 4 * 65536); \
    wslot = (wslot + SLOTB) & (RING * SLOTB - 1); } while (0)

  const unsigned dst = (rank + 1u + (unsigned)(bid >> 5)) & 7u, rg = (unsigned)bid & 31u;
  const unsigned vlin = (unsigned)lane * 16u;
  const char* const xs = g.x + (size_t)(dst * MR + rg * 16u) * 1024 - 4096;
  const unsigned vx0 = (unsigned)((lane >> 4) * 1024 + (lane & 15) * 16 + 4096), vx1 = vx0 + 4096u - 1024u;
#define XLOAD(q) do { GLDS2(ldsbase + LDS_XS + (q) * 4096u, vx0, vx1, xs + (q) * 256); GLDS2(ldsbase + LDS_XS + (q) * 4096u + 2048u, vx0, vx1, xs + (q) * 256 + 8192); } while (0)
  ISSUE(0, ""); ap += 64; wp += 1 << 21;
  if (wid == 7) {
    XLOAD(0);
    asm volatile("s_mov_b32 m0, %0\n\ts_nop 0\n\tglobal_load_lds_dword %1, %2 sc1" :: "s"(ldsbase + LDS_POLL), "v"(WS_CALLS + (unsigned)bid * 128u), "s"(wsl) : "memory");
  }
  ISSUE(0, ""); ap += 64; wp += 1 << 21;
  ISSUE(0, ""); ap += 64; wp += 1 << 21;
  unsigned ra, rb[4];
  { const int o = fr * 64 + fq * 16; ra = ldsbase + (unsigned)(wr * 8192 + (o ^ (((o >> 9) & 1) << 5))); }
  { const int q = (lane & 15) >> 2, p = lane & 3, f = q | ((fq & 1) << 2);
#pragma unroll
    for (int n = 0; n < 4; ++n) rb[n] = ldsbase + (unsigned)(16384 + (8 * fq + q) * 512 + (wc >> 1) * 256 + ((((wc & 1) * 4 + n) ^ f) * 32) + 8 * p); }
  unsigned rslot = 0;
  f32x4 acc[8][4];
#pragma unroll
  for (int m = 0; m < 8; ++m)
#pragma unroll
    for (int n = 0; n < 4; ++n) acc[m][n] = (f32x4){0.f, 0.f, 0.f, 0.f};
  bf16x8 At[8], Bf[4];
#define READS() do { _Pragma("unroll") for (int n = 0; n < 4; ++n) { \
      s16x4 lo = __builtin_amdgcn_ds_read_tr16_b64_v4i16((LAS s16x4*)(size_t)(rb[n])); \
      s16x4 hi = __builtin_amdgcn_ds_read_tr16_b64_v4i16((LAS s16x4*)(size_t)(rb[n] + 2048)); \
      Bf[n] = __builtin_shufflevector(lo, hi, 0, 1, 2, 3, 4, 5, 6, 7); } \
    _Pragma("unroll") for (int m = 0; m < 8; ++m) At[m] = *(const LAS bf16x8*)(size_t)(ra + m * 1024); } while (0)
#define ADV() do { const int d_ = (rslot == (RING - 1) * SLOTB) ? -(int)((RING - 1) * SLOTB) : (int)SLOTB; rslot += d_; ra += d_; \
    _Pragma("unroll") for (int n = 0; n < 4; ++n) rb[n] += d_; \
    asm volatile("" : "+v"(ra), "+v"(rb[0]), "+v"(rb[1]), "+v"(rb[2]), "+v"(rb[3])); } while (0)
#define MMA() do { __builtin_amdgcn_s_setprio(1); _Pragma("unroll") for (int m = 0; m < 8; ++m) _Pragma("unroll") for (int n = 0; n < 4; ++n) \
    acc[m][n] = __builtin_amdgcn_mfma_f32_16x16x32_bf16(Bf[n], At[m], acc[m][n], 0, 0, 0); __builtin_amdgcn_s_setprio(0); } while (0)

  asm volatile("" : "+v"(ra), "+v"(rb[0]), "+v"(rb[1]), "+v"(rb[2]), "+v"(rb[3]));

#define SPIN_COUNT() ((void)0)
#define NEXTK1() do { wo = (wo + ((wo & (1u << 21)) ? (15u << 21) : (1u << 21))) & 0x0fffffffu; } while (0)
#define NEXTK2() do { wo += 1u << 21; if ((wo & (15u << 21)) == 0u) wo = (wo + (2u << 21)) & 0x0fffffffu; } while (0)
  const char* Ab = nullptr;
#define MMA_PHASE() do { BAR(); MMA(); BAR(); SCHED(); } while (0)
  unsigned call = 0;

  unsigned wo = ((rank + 1u) & 7u) << 25;
  auto head = [&](auto role) __attribute__((always_inline)) {
    constexpr int ROLE = decltype(role)::value;
    constexpr int NH = ROLE == 6 ? 6 : (ROLE == 4 || ROLE == 5) ? 5 : 4;
#define STAGE_END(n) do { if (ROLE == 0 && LATEWAIT) { WAIT_L0(); BAR(); MMA(); SCHED(); WAIT_V(n); BAR(); SCHED(); } else { WAIT_V(n); WAIT_L0(); MMA_PHASE(); } } while (0)
#define LOCAL_STAGES(n) do { int n_ = (n); asm volatile("" : "+s"(n_)); _Pragma("nounroll") for (; n_ > 0; --n_) { READS(); ISSUE(ROLE, ""); ap += 64; wp += 1 << 21; ADV(); SCHED(); STAGE_END(4 + NH); } } while (0)
#define REMOTE_STAGES(n, HELP) do { int n_ = (n); asm volatile("" : "+s"(n_)); _Pragma("nounroll") for (; n_ > 0; --n_) { wp = Wb + wo; ap = Ab + (wo >> 5); READS(); ISSUE(HELP, APOL_R); NEXTK1(); ADV(); SCHED(); STAGE_END(HELP ? 4 + NH : 8); } } while (0)
#define TO_STAGED(c) do { Ab = wsl + WS_STAGE + ((c) & 1u) * SLOT_BYTES - 4096; arow = 64; \
      _Pragma("unroll") for (int i = 0; i < 2; ++i) voffA[i] = (unsigned)((pm * 256 + (wid * 2 + i) * 16 + (ob >> 6)) * 64 + ((ob >> 4) & 3) * 16 + 4096 - 1024 * i); } while (0)
#pragma unroll
    for (int m = 0; m < 8; m += 2) asm volatile("" : "+v"(acc[m][0]), "+v"(acc[m][1]), "+v"(acc[m][2]), "+v"(acc[m][3]), "+v"(acc[m + 1][0]), "+v"(acc[m + 1][1]), "+v"(acc[m + 1][2]), "+v"(acc[m + 1][3]));
    if (ROLE == 7) WAIT_V(13); else WAIT_V(8);
    BAR();
    if (ROLE != 0) BAR();

    if constexpr (ROLE == 7) {
      char* const wsd = g.ws[dst];
      const unsigned fo = FLAG_OFF + rg * 4u;
      unsigned xrd = ldsbase + LDS_XS + (unsigned)((lane >> 2) * 256 + (lane & 3) * 16);
#define XSTORE(xv) do { asm volatile("global_store_dwordx4 %0, %1, %2" SPOL "\n\ts_nop 1" :: "v"(vlin), "v"(xv), "s"(dstp) : "memory"); dstp += 65536; } while (0)
#define STORE1(off, val, base, POL) asm volatile("s_mov_b64 exec, 1\n\tglobal_store_dword %0, %1, %2" POL "\n\ts_mov_b64 exec, -1" :: "v"(off), "v"(val), "s"(base) : "memory")
      TS(0);
      READS(); XLOAD(1); ADV();
      SCHED(); WAIT_V(8); WAIT_L0(); MMA_PHASE();
      TS(2);
      READS();
      const unsigned cnt = *(const LAS unsigned*)(lds + LDS_POLL);
      const u32x4 x0 = *(const LAS u32x4*)(size_t)xrd, x1 = *(const LAS u32x4*)(size_t)(xrd + 64u);
      XLOAD(2); XLOAD(3);
      ADV(); SCHED(); WAIT_V(12); WAIT_L0();
      call = (unsigned)__builtin_amdgcn_readfirstlane((int)cnt) + 1u;
      char* const reg = wsd + WS_STAGE + (call & 1u) * SLOT_BYTES + (rank << 20);
      char* dstp = reg + rg * 1024u;
      XSTORE(x0); XSTORE(x1);
      STORE1(WS_CALLS + (unsigned)bid * 128u, call, wsl, " sc1");
      *(LAS unsigned*)(lds + LDS_CTL) = call; *(LAS unsigned*)(lds + LDS_CTL + 4) = 0u;
      WAIT_L0(); MMA_PHASE();
#define PUSH_STAGES(from, to) do { int e_ = (to); asm volatile("" : "+s"(e_)); _Pragma("nounroll") for (int s = (from); s < e_; ++s) { TS(2 * s); \
        READS(); \
        const u32x4 xv = *(const LAS u32x4*)(size_t)(xrd + ((unsigned)s >> 2) * 4096u + ((unsigned)s & 3u) * 64u); \
        ADV(); SCHED(); WAIT_L0(); \
        XSTORE(xv); \
        MMA_PHASE(); } } while (0)
      PUSH_STAGES(2, 4);
      WAIT_V(5);
      PUSH_STAGES(4, 5);
      WAIT_V(4);
      STORE1(fo, call * 16u + 2u, reg, " sc0 sc1");
      PUSH_STAGES(5, 16);
#pragma nounroll
      for (int s = 16; s < 19; ++s) { TS(2 * s); READS(); ADV(); SCHED(); WAIT_L0(); MMA_PHASE(); }
      WAIT_V(0);
      STORE1(fo, call * 16u + 16u, reg, " sc0 sc1");
#pragma nounroll
      for (int s = 19; s < 22; ++s) { TS(2 * s); READS(); ADV(); SCHED(); WAIT_L0(); MMA_PHASE(); }
      TO_STAGED(call);
      wslot = (25u % RING) * SLOTB; wo = (((rank + 5u) & 7u) << 25) + (1u << 21);
      REMOTE_STAGES(5, 0);
    } else if constexpr (ROLE == 4) {
      const unsigned pl = (unsigned)lane % 28u;
      unsigned pollo = 0, pollo2 = 0;
      unsigned* const tmo = (unsigned*)(wsl + WS_TMO);
      bool gave_up = false;
#define POLL_ISSUE() asm volatile("s_mov_b32 m0, %0\n\ts_nop 0\n\tglobal_load_lds_dwordx4 %1, %2 sc0 sc1" :: "s"(ldsbase + LDS_POLL), "v"(pollo2), "s"(wsl) : "memory")
#define POLL_CHECK(r) do { const unsigned need_ = call * 16u + (r); u32x4 pf = *(const LAS u32x4*)(lds + LDS_POLL + pl * 16u); \
      if (!flags_ok(pf, need_)) { const unsigned long long t0_ = __builtin_amdgcn_s_memrealtime(); \
        for (;;) { __builtin_amdgcn_s_sleep(2); SPIN_COUNT(); \
          asm volatile("global_load_dwordx4 %0, %1, %2 sc0 sc1\n\ts_waitcnt vmcnt(0)" : "=&v"(pf) : "v"(pollo), "s"(wsl) : "memory"); \
          if (flags_ok(pf, need_)) break; \
          if ((unsigned)__builtin_amdgcn_readfirstlane((int)__hip_atomic_load(tmo, __ATOMIC_RELAXED, __HIP_MEMORY_SCOPE_AGENT)) == call \
              || __builtin_amdgcn_s_memrealtime() - t0_ > DEADLINE_TICKS) { \
            if (lane == 0) __hip_atomic_store(tmo, call, __ATOMIC_RELAXED, __HIP_MEMORY_SCOPE_AGENT); gave_up = true; break; } } } } while (0)
      LOCAL_STAGES(9);
      call = (unsigned)__builtin_amdgcn_readfirstlane((int)*(const LAS unsigned*)(lds + LDS_CTL));
      pollo = WS_STAGE + (call & 1u) * SLOT_BYTES + (((rank + 1u + (pl >> 2)) & 7u) << 20) + FLAG_OFF + (unsigned)pm * 64u + (pl & 3u) * 16u;
      pollo2 = lane == 63 ? WS_TMO : pollo;
      LOCAL_STAGES(1);
      POLL_ISSUE();
      LOCAL_STAGES(2);
      POLL_CHECK(2);
      LOCAL_STAGES(1);
      TO_STAGED(call);
      REMOTE_STAGES(9, ROLE);
      REMOTE_STAGES(2, 0);
      POLL_ISSUE();
      REMOTE_STAGES(2, 0);
      POLL_CHECK(16);
      if (gave_up || *(const LAS unsigned*)(lds + LDS_POLL + 63 * 16) == call) *(LAS unsigned*)(lds + LDS_CTL + 4) = 1u;
      REMOTE_STAGES(1, 0);
    } else {
      LOCAL_STAGES(13);
      TO_STAGED((unsigned)__builtin_amdgcn_readfirstlane((int)*(const LAS unsigned*)(lds + LDS_CTL)));
      if constexpr (ROLE != 0) { REMOTE_STAGES(9, ROLE); REMOTE_STAGES(5, 0); } else REMOTE_STAGES(14, 0);
    }
  };
  if (wid == 7) head(std::integral_constant<int, 7>{});
  else if (wid == 6) head(std::integral_constant<int, 6>{});
  else if (wid == 5) head(std::integral_constant<int, 5>{});
  else if (wid == 4) head(std::integral_constant<int, 4>{});
  else head(std::integral_constant<int, 0>{});

  wo = (((rank + 1u) & 7u) << 25) + (2u << 21);
  wslot = (30u % RING) * SLOTB; rslot = (27u % RING) * SLOTB;
  for (int s = 27; s < 125; ++s) {
    wp = Wb + wo; ap = Ab + (wo >> 5);
    READS(); ISSUE(0, APOL_R); NEXTK2(); ADV();
    SCHED(); WAIT_V(8); WAIT_L0(); MMA_PHASE();
  }
  READS(); ADV(); SCHED(); WAIT_V(4); WAIT_L0(); MMA_PHASE();
  READS(); ADV(); SCHED(); WAIT_V(0); WAIT_L0(); MMA_PHASE();
  READS(); SCHED(); WAIT_L0(); MMA_PHASE();
  if (wr == 0) BAR();

  {
    const bool bad = __builtin_amdgcn_readfirstlane((int)*(const LAS unsigned*)(lds + LDS_CTL + 4)) != 0;
    const __amdgpu_buffer_rsrc_t orsrc = __builtin_amdgcn_make_buffer_rsrc(g.out, 0, MR * NN * 2, 0x00020000);
    LAS char* wl = lds + wid * 16384;
    const unsigned ooff = (unsigned)(((pm * 256 + wr * 128 + (lane >> 3)) * NN + pn * 256 + wc * 64 + (lane & 7) * 8) * 2);
    if (bad) {
      for (int i = 0; i < 16; ++i) __builtin_amdgcn_raw_buffer_store_b128((u32x4){0x7fc07fc0u, 0x7fc07fc0u, 0x7fc07fc0u, 0x7fc07fc0u}, orsrc, ooff + (unsigned)(i * 8) * (NN * 2), 0, 0);
    } else
#pragma unroll
    for (int m = 0; m < 8; ++m) {
#pragma unroll
      for (int np = 0; np < 2; ++np) {
        const f32x4 x = acc[m][2 * np], y = acc[m][2 * np + 1];
        const unsigned x0 = pk_bf16(silu(x[0]), silu(x[1])), x1 = pk_bf16(silu(x[2]), silu(x[3]));
        const unsigned y0 = pk_bf16(silu(y[0]), silu(y[1])), y1 = pk_bf16(silu(y[2]), silu(y[3]));
        const auto s0 = __builtin_amdgcn_permlane16_swap(x0, y0, false, false);
        const auto s1 = __builtin_amdgcn_permlane16_swap(x1, y1, false, false);
        const u32x4 w = {s0[0], s1[0], s0[1], s1[1]};
        const int c = np * 4 + (fq & 1) * 2 + (fq >> 1);
        *(LAS u32x4*)(wl + (m * 16 + fr) * 128 + ((c ^ (fr & 7)) * 16)) = w;
      }
#pragma unroll
      for (int h = 0; h < 2; ++h) {
        const int r = m * 16 + h * 8 + (lane >> 3);
        const u32x4 w = *(LAS u32x4*)(wl + r * 128 + (((lane & 7) ^ (r & 7)) * 16));
        __builtin_amdgcn_raw_buffer_store_b128(w, orsrc, ooff + (unsigned)(m * 16 + h * 8) * (NN * 2), 0,   2);
      }
    }
  }
}

}

extern "C" void kernel_launch(int rank, int nranks, ncclComm_t comm,
                              hipStream_t stream,
                              void* const* d_in, const int* in_sizes, int n_in,
                              void* d_out, int out_size,
                              const int* in_dtypes, int out_dtype,
                              void* const* d_out_all,
                              void* const* d_ws_all, size_t ws_bytes) {
  (void)comm; (void)in_dtypes; (void)out_dtype; (void)d_out_all;
  if (nranks != NRANK || (unsigned)rank >= (unsigned)NRANK || n_in != 2 || in_sizes[0] != NRANK * MR * KS || in_sizes[1] != NRANK * KS * NN ||
      out_size != MR * NN || ws_bytes < WS_NEED) {
    fprintf(stderr, "kernel_launch: unexpected shapes\n");
    return;
  }
  Params p;
  p.x = (const char*)d_in[0]; p.w = (const char*)d_in[1]; p.out = (char*)d_out; p.rank = rank; p.stamps = nullptr;
  for (int i = 0; i < NRANK; ++i) p.ws[i] = (char*)d_ws_all[i];
  hipLaunchKernelGGL(a2a_gemm_silu, dim3(NWG), dim3(512), 0, stream, p);
  HIP_CHECK(hipGetLastError());
}
```

```cpp
#include <hip/hip_runtime.h>
#include <rccl/rccl.h>
#include <cstdio>
#include <type_traits>

#define HIP_CHECK(expr) do { hipError_t _e = (expr); if (_e != hipSuccess) { \
    fprintf(stderr, "HIP error %s at %s:%d\n", hipGetErrorString(_e), __FILE__, __LINE__); \
    return; } } while (0)

namespace {

#define LAS __attribute__((address_space(3)))
typedef short bf16x8 __attribute__((ext_vector_type(8)));
typedef short s16x4 __attribute__((ext_vector_type(4)));
typedef float f32x4 __attribute__((ext_vector_type(4)));
typedef unsigned u32x4 __attribute__((ext_vector_type(4)));
typedef float f32x2 __attribute__((ext_vector_type(2)));
typedef __bf16 bf16x2_t __attribute__((ext_vector_type(2)));

constexpr int NRANK = 8, MR = 512, KS = 512, NN = 32768;
constexpr int NWG = 256;
constexpr unsigned WS_TMO = 0;
constexpr unsigned WS_CALLS = 65536;
constexpr unsigned WS_STAGE = 2u << 20, SLOT_BYTES = 8u << 20, FLAG_OFF = 32768;
constexpr size_t WS_NEED = (size_t)WS_STAGE + 2 * (size_t)SLOT_BYTES;
constexpr unsigned RING = 4, SLOTB = 32768;
constexpr unsigned LDS_XS = RING * SLOTB;
constexpr unsigned LDS_POLL = LDS_XS + 16384;
constexpr unsigned LDS_CTL = LDS_POLL + 1024;
constexpr unsigned LDS_BYTES = LDS_CTL + 16;
#define TS(i) ((void)0)
#define APOL_R " sc0 sc1"
#define LATEWAIT 1
#define WPOL " nt"
#define OAUX 2
#define SPOL " sc0 sc1 nt"
constexpr unsigned long long DEADLINE_TICKS = 100000000ull;

struct Params { const char* x; const char* w; char* out; char* ws[NRANK]; int rank; unsigned long long* stamps; };

#define WAIT_V(n) asm volatile("s_waitcnt vmcnt(%0)" :: "n"(n) : "memory")
#define WAIT_L0() do { asm volatile("" ::: "memory"); __builtin_amdgcn_s_waitcnt(0xC07F); asm volatile("" ::: "memory"); } while (0)
#define BAR() do { asm volatile("" ::: "memory"); __builtin_amdgcn_s_barrier(); asm volatile("" ::: "memory"); } while (0)
#define SCHED() __builtin_amdgcn_sched_barrier(0)
#define GLDS2P(POL, m0v, v0, v1, base) asm volatile("s_mov_b32 m0, %0\n\ts_nop 0\n\tglobal_load_lds_dwordx4 %1, %3" POL "\n\tglobal_load_lds_dwordx4 %2, %3 offset:1024" POL \
    :: "s"(m0v), "v"(v0), "v"(v1), "s"(base) : "memory")
#define GLDS2(m0v, v0, v1, base) GLDS2P("", m0v, v0, v1, base)

__device__ __forceinline__ unsigned pk_bf16(float lo, float hi) { f32x2 v = {lo, hi}; bf16x2_t b = __builtin_convertvector(v, bf16x2_t); return __builtin_bit_cast(unsigned, b); }
__device__ __forceinline__ float silu(float y) {
  const float c = __builtin_amdgcn_fmed3f(y, -60.f, 60.f);
  const float e = __builtin_amdgcn_exp2f(c * -1.44269504088896340736f);
  return y * __builtin_amdgcn_rcpf(1.0f + e);
}
__device__ __forceinline__ bool flags_ok(const u32x4 f, unsigned need) {
  const int m = min(min((int)(f[0] - need), (int)(f[1] - need)), min((int)(f[2] - need), (int)(f[3] - need)));
  return __builtin_amdgcn_ballot_w64(m < 0) == 0ull;
}

__global__ __launch_bounds__(512, 2) void a2a_gemm_silu(Params g) {
  __shared__ __attribute__((aligned(1024))) char smem[LDS_BYTES];
  LAS char* lds = (LAS char*)smem;
  const int tid = threadIdx.x, wid = __builtin_amdgcn_readfirstlane(tid >> 6), lane = tid & 63;
  const int wr = wid >> 2, wc = wid & 3, fr = lane & 15, fq = lane >> 4;
  const int bid = blockIdx.x, pn = (bid & 7) * 16 + (bid >> 4), pm = (bid >> 3) & 1;
  const unsigned rank = (unsigned)g.rank & 7u;
  char* const wsl = g.ws[rank];
  const unsigned ldsbase = (unsigned)(size_t)lds;

  unsigned voffA[2], voffB[2];
  const int obp = lane * 16, ob = obp ^ (((obp >> 9) & 1) << 5);
#pragma unroll
  for (int i = 0; i < 2; ++i) voffA[i] = (unsigned)((pm * 256 + (wid * 2 + i) * 16 + (ob >> 6)) * 1024 + ((ob >> 4) & 3) * 16 + 4096 - 1024 * i);
#pragma unroll
  for (int i = 0; i < 2; ++i) { const int kr = (wid * 2 + i) * 2 + (lane >> 5), f = (kr & 3) | (((kr >> 3) & 1) << 2), c = (lane & 31) ^ (f << 1);
    voffB[i] = (unsigned)(kr * 65536 + c * 16 + 4096 - 1024 * i); }
  const unsigned m0w = ldsbase + (unsigned)wid * 2048u;
  const char* const Wb = g.w + (size_t)pn * 512 - 4096;
  const char* ap = g.x + (size_t)rank * (MR * 1024) - 4096;
  const char* wp = Wb + ((size_t)(rank * 16u) << 21);
  unsigned wslot = 0;
  unsigned arow = 1024;
#define GLDS1P(POL, m0v, v0, base) asm volatile("s_mov_b32 m0, %0\n\ts_nop 0\n\tglobal_load_lds_dwordx4 %1, %2" POL :: "s"(m0v), "v"(v0), "s"(base) : "memory")
#define ISSUE(HELP, APOL) do { GLDS2P(APOL, m0w + wslot, voffA[0], voffA[1], ap); GLDS2P(WPOL, m0w + wslot + 16384u, voffB[0], voffB[1], wp); \
    if (HELP == 4) GLDS1P(APOL, m0w + wslot + 6144u, voffA[0], ap + 96u * arow); \
    if (HELP == 5) GLDS1P(APOL, m0w + wslot + 5120u, voffA[0], ap + 80u * arow); \
    if (HELP == 6) GLDS2P(WPOL, m0w + wslot + 2048u + 16384u, voffB[0], voffB[1], wp + 4 * 65536); \
    wslot = (wslot + SLOTB) & (RING * SLOTB - 1); } while (0)

  const unsigned dst = (rank + 1u + (unsigned)(bid >> 5)) & 7u, rg = (unsigned)bid & 31u;
  const unsigned vlin = (unsigned)lane * 16u;
  const char* const xs = g.x + (size_t)(dst * MR + rg * 16u) * 1024 - 4096;
  const unsigned vx0 = (unsigned)((lane >> 4) * 1024 + (lane & 15) * 16 + 4096), vx1 = vx0 + 4096u - 1024u;
#define XLOAD(q) do { GLDS2(ldsbase + LDS_XS + (q) * 4096u, vx0, vx1, xs + (q) * 256); GLDS2(ldsbase + LDS_XS + (q) * 4096u + 2048u, vx0, vx1, xs + (q) * 256 + 8192); } while (0)
  ISSUE(0, ""); ap += 64; wp += 1 << 21;
  if (wid == 7) {
    XLOAD(0);
    asm volatile("s_mov_b32 m0, %0\n\ts_nop 0\n\tglobal_load_lds_dword %1, %2 sc1" :: "s"(ldsbase + LDS_POLL), "v"(WS_CALLS + (unsigned)bid * 128u), "s"(wsl) : "memory");
  }
  ISSUE(0, ""); ap += 64; wp += 1 << 21;
  ISSUE(0, ""); ap += 64; wp += 1 << 21;
  unsigned ra, rb[4];
  { const int o = fr * 64 + fq * 16; ra = ldsbase + (unsigned)(wr * 8192 + (o ^ (((o >> 9) & 1) << 5))); }
  { const int q = (lane & 15) >> 2, p = lane & 3, f = q | ((fq & 1) << 2);
#pragma unroll
    for (int n = 0; n < 4; ++n) rb[n] = ldsbase + (unsigned)(16384 + (8 * fq + q) * 512 + (wc >> 1) * 256 + ((((wc & 1) * 4 + n) ^ f) * 32) + 8 * p); }
  unsigned rslot = 0;
  f32x4 acc[8][4];
#pragma unroll
  for (int m = 0; m < 8; ++m)
#pragma unroll
    for (int n = 0; n < 4; ++n) acc[m][n] = (f32x4){0.f, 0.f, 0.f, 0.f};
  bf16x8 At[8], Bf[4];
#define READS() do { _Pragma("unroll") for (int n = 0; n < 4; ++n) { \
      s16x4 lo = __builtin_amdgcn_ds_read_tr16_b64_v4i16((LAS s16x4*)(size_t)(rb[n])); \
      s16x4 hi = __builtin_amdgcn_ds_read_tr16_b64_v4i16((LAS s16x4*)(size_t)(rb[n] + 2048)); \
      Bf[n] = __builtin_shufflevector(lo, hi, 0, 1, 2, 3, 4, 5, 6, 7); } \
    _Pragma("unroll") for (int m = 0; m < 8; ++m) At[m] = *(const LAS bf16x8*)(size_t)(ra + m * 1024); } while (0)
#define ADV() do { const int d_ = (rslot == (RING - 1) * SLOTB) ? -(int)((RING - 1) * SLOTB) : (int)SLOTB; rslot += d_; ra += d_; \
    _Pragma("unroll") for (int n = 0; n < 4; ++n) rb[n] += d_; \
    asm volatile("" : "+v"(ra), "+v"(rb[0]), "+v"(rb[1]), "+v"(rb[2]), "+v"(rb[3])); } while (0)
#define MMA() do { __builtin_amdgcn_s_setprio(1); _Pragma("unroll") for (int m = 0; m < 8; ++m) _Pragma("unroll") for (int n = 0; n < 4; ++n) \
    acc[m][n] = __builtin_amdgcn_mfma_f32_16x16x32_bf16(Bf[n], At[m], acc[m][n], 0, 0, 0); __builtin_amdgcn_s_setprio(0); } while (0)

  asm volatile("" : "+v"(ra), "+v"(rb[0]), "+v"(rb[1]), "+v"(rb[2]), "+v"(rb[3]));

#define SPIN_COUNT() ((void)0)
#define NEXTK1() do { wo = (wo + ((wo & (1u << 21)) ? (15u << 21) : (1u << 21))) & 0x0fffffffu; } while (0)
#define NEXTK2() do { wo += 1u << 21; if ((wo & (15u << 21)) == 0u) wo = (wo + (2u << 21)) & 0x0fffffffu; } while (0)
  const char* Ab = nullptr;
#define MMA_PHASE() do { BAR(); MMA(); BAR(); SCHED(); } while (0)
  unsigned call = 0;

  unsigned wo = ((rank + 1u) & 7u) << 25;
  auto head = [&](auto role) __attribute__((always_inline)) {
    constexpr int ROLE = decltype(role)::value;
    constexpr int NH = ROLE == 6 ? 6 : (ROLE == 4 || ROLE == 5) ? 5 : 4;
#define STAGE_END(n) do { if (ROLE == 0 && LATEWAIT) { WAIT_L0(); BAR(); MMA(); SCHED(); WAIT_V(n); BAR(); SCHED(); } else { WAIT_V(n); WAIT_L0(); MMA_PHASE(); } } while (0)
#define LOCAL_STAGES(n) do { int n_ = (n); asm volatile("" : "+s"(n_)); _Pragma("nounroll") for (; n_ > 0; --n_) { READS(); ISSUE(ROLE, ""); ap += 64; wp += 1 << 21; ADV(); SCHED(); STAGE_END(4 + NH); } } while (0)
#define REMOTE_STAGES(n, HELP) do { int n_ = (n); asm volatile("" : "+s"(n_)); _Pragma("nounroll") for (; n_ > 0; --n_) { wp = Wb + wo; ap = Ab + (wo >> 5); READS(); ISSUE(HELP, APOL_R); NEXTK1(); ADV(); SCHED(); STAGE_END(HELP ? 4 + NH : 8); } } while (0)
#define TO_STAGED(c) do { Ab = wsl + WS_STAGE + ((c) & 1u) * SLOT_BYTES - 4096; arow = 64; \
      _Pragma("unroll") for (int i = 0; i < 2; ++i) voffA[i] = (unsigned)((pm * 256 + (wid * 2 + i) * 16 + (ob >> 6)) * 64 + ((ob >> 4) & 3) * 16 + 4096 - 1024 * i); } while (0)
#pragma unroll
    for (int m = 0; m < 8; m += 2) asm volatile("" : "+v"(acc[m][0]), "+v"(acc[m][1]), "+v"(acc[m][2]), "+v"(acc[m][3]), "+v"(acc[m + 1][0]), "+v"(acc[m + 1][1]), "+v"(acc[m + 1][2]), "+v"(acc[m + 1][3]));
    if (ROLE == 7) WAIT_V(13); else WAIT_V(8);
    BAR();
    if (ROLE != 0) BAR();

    if constexpr (ROLE == 7) {
      char* const wsd = g.ws[dst];
      const unsigned fo = FLAG_OFF + rg * 4u;
      unsigned xrd = ldsbase + LDS_XS + (unsigned)((lane >> 2) * 256 + (lane & 3) * 16);
#define XSTORE(xv) do { asm volatile("global_store_dwordx4 %0, %1, %2" SPOL "\n\ts_nop 1" :: "v"(vlin), "v"(xv), "s"(dstp) : "memory"); dstp += 65536; } while (0)
#define STORE1(off, val, base, POL) asm volatile("s_mov_b64 exec, 1\n\tglobal_store_dword %0, %1, %2" POL "\n\ts_mov_b64 exec, -1" :: "v"(off), "v"(val), "s"(base) : "memory")
      TS(0);
      READS(); XLOAD(1); ADV();
      SCHED(); WAIT_V(8); WAIT_L0(); MMA_PHASE();
      TS(2);
      READS();
      const unsigned cnt = *(const LAS unsigned*)(lds + LDS_POLL);
      const u32x4 x0 = *(const LAS u32x4*)(size_t)xrd, x1 = *(const LAS u32x4*)(size_t)(xrd + 64u);
      XLOAD(2); XLOAD(3);
      ADV(); SCHED(); WAIT_V(12); WAIT_L0();
      call = (unsigned)__builtin_amdgcn_readfirstlane((int)cnt) + 1u;
      char* const reg = wsd + WS_STAGE + (call & 1u) * SLOT_BYTES + (rank << 20);
      char* dstp = reg + rg * 1024u;
      XSTORE(x0); XSTORE(x1);
      STORE1(WS_CALLS + (unsigned)bid * 128u, call, wsl, " sc1");
      *(LAS unsigned*)(lds + LDS_CTL) = call; *(LAS unsigned*)(lds + LDS_CTL + 4) = 0u;
      WAIT_L0(); MMA_PHASE();
#define PUSH_STAGES(from, to) do { int e_ = (to); asm volatile("" : "+s"(e_)); _Pragma("nounroll") for (int s = (from); s < e_; ++s) { TS(2 * s); \
        READS(); \
        const u32x4 xv = *(const LAS u32x4*)(size_t)(xrd + ((unsigned)s >> 2) * 4096u + ((unsigned)s & 3u) * 64u); \
        ADV(); SCHED(); WAIT_L0(); \
        XSTORE(xv); \
        MMA_PHASE(); } } while (0)
      PUSH_STAGES(2, 4);
      WAIT_V(5);
      PUSH_STAGES(4, 5);
      WAIT_V(4);
      STORE1(fo, call * 16u + 2u, reg, " sc0 sc1");
      PUSH_STAGES(5, 16);
#pragma nounroll
      for (int s = 16; s < 19; ++s) { TS(2 * s); READS(); ADV(); SCHED(); WAIT_L0(); MMA_PHASE(); }
      WAIT_V(0);
      STORE1(fo, call * 16u + 16u, reg, " sc0 sc1");
#pragma nounroll
      for (int s = 19; s < 22; ++s) { TS(2 * s); READS(); ADV(); SCHED(); WAIT_L0(); MMA_PHASE(); }
      TO_STAGED(call);
      wslot = (25u % RING) * SLOTB; wo = (((rank + 5u) & 7u) << 25) + (1u << 21);
      REMOTE_STAGES(5, 0);
    } else if constexpr (ROLE == 4) {
      const unsigned pl = (unsigned)lane % 28u;
      unsigned pollo = 0, pollo2 = 0;
      unsigned* const tmo = (unsigned*)(wsl + WS_TMO);
      bool gave_up = false;
#define POLL_ISSUE() asm volatile("s_mov_b32 m0, %0\n\ts_nop 0\n\tglobal_load_lds_dwordx4 %1, %2 sc0 sc1" :: "s"(ldsbase + LDS_POLL), "v"(pollo2), "s"(wsl) : "memory")
#define POLL_CHECK(r) do { const unsigned need_ = call * 16u + (r); u32x4 pf = *(const LAS u32x4*)(lds + LDS_POLL + pl * 16u); \
      if (!flags_ok(pf, need_)) { const unsigned long long t0_ = __builtin_amdgcn_s_memrealtime(); \
        for (;;) { __builtin_amdgcn_s_sleep(2); SPIN_COUNT(); \
          asm volatile("global_load_dwordx4 %0, %1, %2 sc0 sc1\n\ts_waitcnt vmcnt(0)" : "=&v"(pf) : "v"(pollo), "s"(wsl) : "memory"); \
          if (flags_ok(pf, need_)) break; \
          if ((unsigned)__builtin_amdgcn_readfirstlane((int)__hip_atomic_load(tmo, __ATOMIC_RELAXED, __HIP_MEMORY_SCOPE_AGENT)) == call \
              || __builtin_amdgcn_s_memrealtime() - t0_ > DEADLINE_TICKS) { \
            if (lane == 0) __hip_atomic_store(tmo, call, __ATOMIC_RELAXED, __HIP_MEMORY_SCOPE_AGENT); gave_up = true; break; } } } } while (0)
      LOCAL_STAGES(9);
      call = (unsigned)__builtin_amdgcn_readfirstlane((int)*(const LAS unsigned*)(lds + LDS_CTL));
      pollo = WS_STAGE + (call & 1u) * SLOT_BYTES + (((rank + 1u + (pl >> 2)) & 7u) << 20) + FLAG_OFF + (unsigned)pm * 64u + (pl & 3u) * 16u;
      pollo2 = lane == 63 ? WS_TMO : pollo;
      LOCAL_STAGES(1);
      POLL_ISSUE();
      LOCAL_STAGES(2);
      POLL_CHECK(2);
      LOCAL_STAGES(1);
      TO_STAGED(call);
      REMOTE_STAGES(9, ROLE);
      REMOTE_STAGES(2, 0);
      POLL_ISSUE();
      REMOTE_STAGES(2, 0);
      POLL_CHECK(16);
      if (gave_up || *(const LAS unsigned*)(lds + LDS_POLL + 63 * 16) == call) *(LAS unsigned*)(lds + LDS_CTL + 4) = 1u;
      REMOTE_STAGES(1, 0);
    } else {
      LOCAL_STAGES(13);
      TO_STAGED((unsigned)__builtin_amdgcn_readfirstlane((int)*(const LAS unsigned*)(lds + LDS_CTL)));
      if constexpr (ROLE != 0) { REMOTE_STAGES(9, ROLE); REMOTE_STAGES(5, 0); } else REMOTE_STAGES(14, 0);
    }
  };
  if (wid == 7) head(std::integral_constant<int, 7>{});
  else if (wid == 6) head(std::integral_constant<int, 6>{});
  else if (wid == 5) head(std::integral_constant<int, 5>{});
  else if (wid == 4) head(std::integral_constant<int, 4>{});
  else head(std::integral_constant<int, 0>{});

  wo = (((rank + 1u) & 7u) << 25) + (2u << 21);
  wslot = (30u % RING) * SLOTB; rslot = (27u % RING) * SLOTB;
  for (int s = 27; s < 125; ++s) {
    wp = Wb + wo; ap = Ab + (wo >> 5);
    READS(); ISSUE(0, APOL_R); NEXTK2(); ADV();
    SCHED(); WAIT_V(8); WAIT_L0(); MMA_PHASE();
  }
  READS(); ADV(); SCHED(); WAIT_V(4); WAIT_L0(); MMA_PHASE();
  READS(); ADV(); SCHED(); WAIT_V(0); WAIT_L0(); MMA_PHASE();
  READS(); SCHED(); WAIT_L0(); MMA_PHASE();
  if (wr == 0) BAR();

  {
    const bool bad = __builtin_amdgcn_readfirstlane((int)*(const LAS unsigned*)(lds + LDS_CTL + 4)) != 0;
    const __amdgpu_buffer_rsrc_t orsrc = __builtin_amdgcn_make_buffer_rsrc(g.out, 0, MR * NN * 2, 0x00020000);
    LAS char* wl = lds + wid * 16384;
    const unsigned ooff = (unsigned)(((pm * 256 + wr * 128 + (lane >> 3)) * NN + pn * 256 + wc * 64 + (lane & 7) * 8) * 2);
    if (bad) {
      for (int i = 0; i < 16; ++i) __builtin_amdgcn_raw_buffer_store_b128((u32x4){0x7fc07fc0u, 0x7fc07fc0u, 0x7fc07fc0u, 0x7fc07fc0u}, orsrc, ooff + (unsigned)(i * 8) * (NN * 2), 0, 0);
    } else
#pragma unroll
    for (int m = 0; m < 8; ++m) {
#pragma unroll
      for (int np = 0; np < 2; ++np) {
        const f32x4 x = acc[m][2 * np], y = acc[m][2 * np + 1];
        const unsigned x0 = pk_bf16(silu(x[0]), silu(x[1])), x1 = pk_bf16(silu(x[2]), silu(x[3]));
        const unsigned y0 = pk_bf16(silu(y[0]), silu(y[1])), y1 = pk_bf16(silu(y[2]), silu(y[3]));
        const auto s0 = __builtin_amdgcn_permlane16_swap(x0, y0, false, false);
        const auto s1 = __builtin_amdgcn_permlane16_swap(x1, y1, false, false);
        const u32x4 w = {s0[0], s1[0], s0[1], s1[1]};
        const int c = np * 4 + (fq & 1) * 2 + (fq >> 1);
        *(LAS u32x4*)(wl + (m * 16 + fr) * 128 + ((c ^ (fr & 7)) * 16)) = w;
      }
#pragma unroll
      for (int h = 0; h < 2; ++h) {
        const int r = m * 16 + h * 8 + (lane >> 3);
        const u32x4 w = *(LAS u32x4*)(wl + r * 128 + (((lane & 7) ^ (r & 7)) * 16));
        __builtin_amdgcn_raw_buffer_store_b128(w, orsrc, ooff + (unsigned)(m * 16 + h * 8) * (NN * 2), 0, OAUX);
      }
    }
  }
}

}

extern "C" void kernel_launch(int rank, int nranks, ncclComm_t comm,
                              hipStream_t stream,
                              void* const* d_in, const int* in_sizes, int n_in,
                              void* d_out, int out_size,
                              const int* in_dtypes, int out_dtype,
                              void* const* d_out_all,
                              void* const* d_ws_all, size_t ws_bytes) {
  (void)comm; (void)in_dtypes; (void)out_dtype; (void)d_out_all;
  if (nranks != NRANK || (unsigned)rank >= (unsigned)NRANK || n_in != 2 || in_sizes[0] != NRANK * MR * KS || in_sizes[1] != NRANK * KS * NN ||
      out_size != MR * NN || ws_bytes < WS_NEED) {
    fprintf(stderr, "kernel_launch: unexpected shapes\n");
    return;
  }
  Params p;
  p.x = (const char*)d_in[0]; p.w = (const char*)d_in[1]; p.out = (char*)d_out; p.rank = rank; p.stamps = nullptr;
  for (int i = 0; i < NRANK; ++i) p.ws[i] = (char*)d_ws_all[i];
  hipLaunchKernelGGL(a2a_gemm_silu, dim3(NWG), dim3(512), 0, stream, p);
  HIP_CHECK(hipGetLastError());
}
```

```cpp
#include <hip/hip_runtime.h>
#include <rccl/rccl.h>
#include <cstdio>
#include <type_traits>

#define HIP_CHECK(expr) do { hipError_t _e = (expr); if (_e != hipSuccess) { \
    fprintf(stderr, "HIP error %s at %s:%d\n", hipGetErrorString(_e), __FILE__, __LINE__); \
    return; } } while (0)

namespace {

#define LAS __attribute__((address_space(3)))
typedef short bf16x8 __attribute__((ext_vector_type(8)));
typedef short s16x4 __attribute__((ext_vector_type(4)));
typedef float f32x4 __attribute__((ext_vector_type(4)));
typedef unsigned u32x4 __attribute__((ext_vector_type(4)));
typedef float f32x2 __attribute__((ext_vector_type(2)));
typedef __bf16 bf16x2_t __attribute__((ext_vector_type(2)));

constexpr int NRANK = 8, MR = 512, KS = 512, NN = 32768;
constexpr int NWG = 256;
constexpr unsigned WS_TMO = 0;
constexpr unsigned WS_CALLS = 65536;
constexpr unsigned WS_STAGE = 2u << 20, SLOT_BYTES = 8u << 20, FLAG_OFF = 32768;
constexpr size_t WS_NEED = (size_t)WS_STAGE + 2 * (size_t)SLOT_BYTES;
constexpr unsigned RING = 4, SLOTB = 32768;
constexpr unsigned LDS_XS = RING * SLOTB;
constexpr unsigned LDS_POLL = LDS_XS + 16384;
constexpr unsigned LDS_CTL = LDS_POLL + 1024;
constexpr unsigned LDS_BYTES = LDS_CTL + 16;
#define TS(i) ((void)0)
#define APOL_R " sc0 sc1"
#define LATEWAIT 1
#define WPOL " nt"
#define OAUX 2
#define PACE_H1 10
#define PACE_H2 3
#define PACE_M1 0
#define PACE_M2 0
#define PACE_END 27
#define SPOL " sc0 sc1 nt"
constexpr unsigned long long DEADLINE_TICKS = 100000000ull;

struct Params { const char* x; const char* w; char* out; char* ws[NRANK]; int rank; unsigned long long* stamps; };

#define WAIT_V(n) asm volatile("s_waitcnt vmcnt(%0)" :: "n"(n) : "memory")
#define WAIT_L0() do { asm volatile("" ::: "memory"); __builtin_amdgcn_s_waitcnt(0xC07F); asm volatile("" ::: "memory"); } while (0)
#define BAR() do { asm volatile("" ::: "memory"); __builtin_amdgcn_s_barrier(); asm volatile("" ::: "memory"); } while (0)
#define SCHED() __builtin_amdgcn_sched_barrier(0)
#define GLDS2P(POL, m0v, v0, v1, base) asm volatile("s_mov_b32 m0, %0\n\ts_nop 0\n\tglobal_load_lds_dwordx4 %1, %3" POL "\n\tglobal_load_lds_dwordx4 %2, %3 offset:1024" POL \
    :: "s"(m0v), "v"(v0), "v"(v1), "s"(base) : "memory")
#define GLDS2(m0v, v0, v1, base) GLDS2P("", m0v, v0, v1, base)

__device__ __forceinline__ unsigned pk_bf16(float lo, float hi) { f32x2 v = {lo, hi}; bf16x2_t b = __builtin_convertvector(v, bf16x2_t); return __builtin_bit_cast(unsigned, b); }
__device__ __forceinline__ float silu(float y) {
  const float c = __builtin_amdgcn_fmed3f(y, -60.f, 60.f);
  const float e = __builtin_amdgcn_exp2f(c * -1.44269504088896340736f);
  return y * __builtin_amdgcn_rcpf(1.0f + e);
}
__device__ __forceinline__ bool flags_ok(const u32x4 f, unsigned need) {
  const int m = min(min((int)(f[0] - need), (int)(f[1] - need)), min((int)(f[2] - need), (int)(f[3] - need)));
  return __builtin_amdgcn_ballot_w64(m < 0) == 0ull;
}

__global__ __launch_bounds__(512, 2) void a2a_gemm_silu(Params g) {
  __shared__ __attribute__((aligned(1024))) char smem[LDS_BYTES];
  LAS char* lds = (LAS char*)smem;
  const int tid = threadIdx.x, wid = __builtin_amdgcn_readfirstlane(tid >> 6), lane = tid & 63;
  const int wr = wid >> 2, wc = wid & 3, fr = lane & 15, fq = lane >> 4;
  const int bid = blockIdx.x, pn = (bid & 7) * 16 + (bid >> 4), pm = (bid >> 3) & 1;
  const unsigned rank = (unsigned)g.rank & 7u;
  char* const wsl = g.ws[rank];
  const unsigned ldsbase = (unsigned)(size_t)lds;

  unsigned voffA[2], voffB[2];
  const int obp = lane * 16, ob = obp ^ (((obp >> 9) & 1) << 5);
#pragma unroll
  for (int i = 0; i < 2; ++i) voffA[i] = (unsigned)((pm * 256 + (wid * 2 + i) * 16 + (ob >> 6)) * 1024 + ((ob >> 4) & 3) * 16 + 4096 - 1024 * i);
#pragma unroll
  for (int i = 0; i < 2; ++i) { const int kr = (wid * 2 + i) * 2 + (lane >> 5), f = (kr & 3) | (((kr >> 3) & 1) << 2), c = (lane & 31) ^ (f << 1);
    voffB[i] = (unsigned)(kr * 65536 + c * 16 + 4096 - 1024 * i); }
  const unsigned m0w = ldsbase + (unsigned)wid * 2048u;
  const char* const Wb = g.w + (size_t)pn * 512 - 4096;
  const char* ap = g.x + (size_t)rank * (MR * 1024) - 4096;
  const char* wp = Wb + ((size_t)(rank * 16u) << 21);
  unsigned wslot = 0;
  unsigned arow = 1024;
#define GLDS1P(POL, m0v, v0, base) asm volatile("s_mov_b32 m0, %0\n\ts_nop 0\n\tglobal_load_lds_dwordx4 %1, %2" POL :: "s"(m0v), "v"(v0), "s"(base) : "memory")
#define ISSUE(HELP, APOL) do { GLDS2P(APOL, m0w + wslot, voffA[0], voffA[1], ap); GLDS2P(WPOL, m0w + wslot + 16384u, voffB[0], voffB[1], wp); \
    if (HELP == 4) GLDS1P(APOL, m0w + wslot + 6144u, voffA[0], ap + 96u * arow); \
    if (HELP == 5) GLDS1P(APOL, m0w + wslot + 5120u, voffA[0], ap + 80u * arow); \
    if (HELP == 6) GLDS2P(WPOL, m0w + wslot + 2048u + 16384u, voffB[0], voffB[1], wp + 4 * 65536); \
    wslot = (wslot + SLOTB) & (RING * SLOTB - 1); } while (0)

  const unsigned dst = (rank + 1u + (unsigned)(bid >> 5)) & 7u, rg = (unsigned)bid & 31u;
  const unsigned vlin = (unsigned)lane * 16u;
  const char* const xs = g.x + (size_t)(dst * MR + rg * 16u) * 1024 - 4096;
  const unsigned vx0 = (unsigned)((lane >> 4) * 1024 + (lane & 15) * 16 + 4096), vx1 = vx0 + 4096u - 1024u;
#define XLOAD(q) do { GLDS2(ldsbase + LDS_XS + (q) * 4096u, vx0, vx1, xs + (q) * 256); GLDS2(ldsbase + LDS_XS + (q) * 4096u + 2048u, vx0, vx1, xs + (q) * 256 + 8192); } while (0)
  ISSUE(0, ""); ap += 64; wp += 1 << 21;
  if (wid == 7) {
    XLOAD(0);
    asm volatile("s_mov_b32 m0, %0\n\ts_nop 0\n\tglobal_load_lds_dword %1, %2 sc1" :: "s"(ldsbase + LDS_POLL), "v"(WS_CALLS + (unsigned)bid * 128u), "s"(wsl) : "memory");
  }
  ISSUE(0, ""); ap += 64; wp += 1 << 21;
  ISSUE(0, ""); ap += 64; wp += 1 << 21;
  unsigned ra, rb[4];
  { const int o = fr * 64 + fq * 16; ra = ldsbase + (unsigned)(wr * 8192 + (o ^ (((o >> 9) & 1) << 5))); }
  { const int q = (lane & 15) >> 2, p = lane & 3, f = q | ((fq & 1) << 2);
#pragma unroll
    for (int n = 0; n < 4; ++n) rb[n] = ldsbase + (unsigned)(16384 + (8 * fq + q) * 512 + (wc >> 1) * 256 + ((((wc & 1) * 4 + n) ^ f) * 32) + 8 * p); }
  unsigned rslot = 0;
  f32x4 acc[8][4];
#pragma unroll
  for (int m = 0; m < 8; ++m)
#pragma unroll
    for (int n = 0; n < 4; ++n) acc[m][n] = (f32x4){0.f, 0.f, 0.f, 0.f};
  bf16x8 At[8], Bf[4];
#define READS() do { _Pragma("unroll") for (int n = 0; n < 4; ++n) { \
      s16x4 lo = __builtin_amdgcn_ds_read_tr16_b64_v4i16((LAS s16x4*)(size_t)(rb[n])); \
      s16x4 hi = __builtin_amdgcn_ds_read_tr16_b64_v4i16((LAS s16x4*)(size_t)(rb[n] + 2048)); \
      Bf[n] = __builtin_shufflevector(lo, hi, 0, 1, 2, 3, 4, 5, 6, 7); } \
    _Pragma("unroll") for (int m = 0; m < 8; ++m) At[m] = *(const LAS bf16x8*)(size_t)(ra + m * 1024); } while (0)
#define ADV() do { const int d_ = (rslot == (RING - 1) * SLOTB) ? -(int)((RING - 1) * SLOTB) : (int)SLOTB; rslot += d_; ra += d_; \
    _Pragma("unroll") for (int n = 0; n < 4; ++n) rb[n] += d_; \
    asm volatile("" : "+v"(ra), "+v"(rb[0]), "+v"(rb[1]), "+v"(rb[2]), "+v"(rb[3])); } while (0)
#define MMA() do { __builtin_amdgcn_s_setprio(1); _Pragma("unroll") for (int m = 0; m < 8; ++m) _Pragma("unroll") for (int n = 0; n < 4; ++n) \
    acc[m][n] = __builtin_amdgcn_mfma_f32_16x16x32_bf16(Bf[n], At[m], acc[m][n], 0, 0, 0); __builtin_amdgcn_s_setprio(0); } while (0)

  asm volatile("" : "+v"(ra), "+v"(rb[0]), "+v"(rb[1]), "+v"(rb[2]), "+v"(rb[3]));

#define SPIN_COUNT() ((void)0)
#define NEXTK1() do { wo = (wo + ((wo & (1u << 21)) ? (15u << 21) : (1u << 21))) & 0x0fffffffu; } while (0)
#define NEXTK2() do { wo += 1u << 21; if ((wo & (15u << 21)) == 0u) wo = (wo + (2u << 21)) & 0x0fffffffu; } while (0)
  const char* Ab = nullptr;
#define MMA_PHASE_P(p) do { BAR(); MMA(); if (p) { SCHED(); __builtin_amdgcn_s_sleep(p); } BAR(); SCHED(); } while (0)
#define MMA_PHASE() MMA_PHASE_P(0)
  unsigned call = 0;

  unsigned wo = ((rank + 1u) & 7u) << 25;
  auto head = [&](auto role) __attribute__((always_inline)) {
    constexpr int ROLE = decltype(role)::value;
    constexpr int NH = ROLE == 6 ? 6 : (ROLE == 4 || ROLE == 5) ? 5 : 4;
#define STAGE_END(n, p) do { if (ROLE == 0 && LATEWAIT) { WAIT_L0(); BAR(); MMA(); SCHED(); if (p) __builtin_amdgcn_s_sleep(p); WAIT_V(n); BAR(); SCHED(); } else { WAIT_V(n); WAIT_L0(); MMA_PHASE(); } } while (0)
#define LOCAL_STAGES(n) do { int n_ = (n); asm volatile("" : "+s"(n_)); _Pragma("nounroll") for (; n_ > 0; --n_) { READS(); ISSUE(ROLE, ""); ap += 64; wp += 1 << 21; ADV(); SCHED(); STAGE_END(4 + NH, PACE_H1); } } while (0)
#define REMOTE_STAGES(n, HELP) do { int n_ = (n); asm volatile("" : "+s"(n_)); _Pragma("nounroll") for (; n_ > 0; --n_) { wp = Wb + wo; ap = Ab + (wo >> 5); READS(); ISSUE(HELP, APOL_R); NEXTK1(); ADV(); SCHED(); STAGE_END(HELP ? 4 + NH : 8, PACE_H2); } } while (0)
#define TO_STAGED(c) do { Ab = wsl + WS_STAGE + ((c) & 1u) * SLOT_BYTES - 4096; arow = 64; \
      _Pragma("unroll") for (int i = 0; i < 2; ++i) voffA[i] = (unsigned)((pm * 256 + (wid * 2 + i) * 16 + (ob >> 6)) * 64 + ((ob >> 4) & 3) * 16 + 4096 - 1024 * i); } while (0)
#pragma unroll
    for (int m = 0; m < 8; m += 2) asm volatile("" : "+v"(acc[m][0]), "+v"(acc[m][1]), "+v"(acc[m][2]), "+v"(acc[m][3]), "+v"(acc[m + 1][0]), "+v"(acc[m + 1][1]), "+v"(acc[m + 1][2]), "+v"(acc[m + 1][3]));
    if (ROLE == 7) WAIT_V(13); else WAIT_V(8);
    BAR();
    if (ROLE != 0) BAR();

    if constexpr (ROLE == 7) {
      char* const wsd = g.ws[dst];
      const unsigned fo = FLAG_OFF + rg * 4u;
      unsigned xrd = ldsbase + LDS_XS + (unsigned)((lane >> 2) * 256 + (lane & 3) * 16);
#define XSTORE(xv) do { asm volatile("global_store_dwordx4 %0, %1, %2" SPOL "\n\ts_nop 1" :: "v"(vlin), "v"(xv), "s"(dstp) : "memory"); dstp += 65536; } while (0)
#define STORE1(off, val, base, POL) asm volatile("s_mov_b64 exec, 1\n\tglobal_store_dword %0, %1, %2" POL "\n\ts_mov_b64 exec, -1" :: "v"(off), "v"(val), "s"(base) : "memory")
      TS(0);
      READS(); XLOAD(1); ADV();
      SCHED(); WAIT_V(8); WAIT_L0(); MMA_PHASE();
      TS(2);
      READS();
      const unsigned cnt = *(const LAS unsigned*)(lds + LDS_POLL);
      const u32x4 x0 = *(const LAS u32x4*)(size_t)xrd, x1 = *(const LAS u32x4*)(size_t)(xrd + 64u);
      XLOAD(2); XLOAD(3);
      ADV(); SCHED(); WAIT_V(12); WAIT_L0();
      call = (unsigned)__builtin_amdgcn_readfirstlane((int)cnt) + 1u;
      char* const reg = wsd + WS_STAGE + (call & 1u) * SLOT_BYTES + (rank << 20);
      char* dstp = reg + rg * 1024u;
      XSTORE(x0); XSTORE(x1);
      STORE1(WS_CALLS + (unsigned)bid * 128u, call, wsl, " sc1");
      *(LAS unsigned*)(lds + LDS_CTL) = call; *(LAS unsigned*)(lds + LDS_CTL + 4) = 0u;
      WAIT_L0(); MMA_PHASE();
#define PUSH_STAGES(from, to) do { int e_ = (to); asm volatile("" : "+s"(e_)); _Pragma("nounroll") for (int s = (from); s < e_; ++s) { TS(2 * s); \
        READS(); \
        const u32x4 xv = *(const LAS u32x4*)(size_t)(xrd + ((unsigned)s >> 2) * 4096u + ((unsigned)s & 3u) * 64u); \
        ADV(); SCHED(); WAIT_L0(); \
        XSTORE(xv); \
        MMA_PHASE(); } } while (0)
      PUSH_STAGES(2, 4);
      WAIT_V(5);
      PUSH_STAGES(4, 5);
      WAIT_V(4);
      STORE1(fo, call * 16u + 2u, reg, " sc0 sc1");
      PUSH_STAGES(5, 16);
#pragma nounroll
      for (int s = 16; s < 19; ++s) { TS(2 * s); READS(); ADV(); SCHED(); WAIT_L0(); MMA_PHASE(); }
      WAIT_V(0);
      STORE1(fo, call * 16u + 16u, reg, " sc0 sc1");
#pragma nounroll
      for (int s = 19; s < 22; ++s) { TS(2 * s); READS(); ADV(); SCHED(); WAIT_L0(); MMA_PHASE(); }
      TO_STAGED(call);
      wslot = (25u % RING) * SLOTB; wo = (((rank + 5u) & 7u) << 25) + (1u << 21);
      REMOTE_STAGES(5, 0);
    } else if constexpr (ROLE == 4) {
      const unsigned pl = (unsigned)lane % 28u;
      unsigned pollo = 0, pollo2 = 0;
      unsigned* const tmo = (unsigned*)(wsl + WS_TMO);
      bool gave_up = false;
#define POLL_ISSUE() asm volatile("s_mov_b32 m0, %0\n\ts_nop 0\n\tglobal_load_lds_dwordx4 %1, %2 sc0 sc1" :: "s"(ldsbase + LDS_POLL), "v"(pollo2), "s"(wsl) : "memory")
#define POLL_CHECK(r) do { const unsigned need_ = call * 16u + (r); u32x4 pf = *(const LAS u32x4*)(lds + LDS_POLL + pl * 16u); \
      if (!flags_ok(pf, need_)) { const unsigned long long t0_ = __builtin_amdgcn_s_memrealtime(); \
        for (;;) { __builtin_amdgcn_s_sleep(2); SPIN_COUNT(); \
          asm volatile("global_load_dwordx4 %0, %1, %2 sc0 sc1\n\ts_waitcnt vmcnt(0)" : "=&v"(pf) : "v"(pollo), "s"(wsl) : "memory"); \
          if (flags_ok(pf, need_)) break; \
          if ((unsigned)__builtin_amdgcn_readfirstlane((int)__hip_atomic_load(tmo, __ATOMIC_RELAXED, __HIP_MEMORY_SCOPE_AGENT)) == call \
              || __builtin_amdgcn_s_memrealtime() - t0_ > DEADLINE_TICKS) { \
            if (lane == 0) __hip_atomic_store(tmo, call, __ATOMIC_RELAXED, __HIP_MEMORY_SCOPE_AGENT); gave_up = true; break; } } } } while (0)
      LOCAL_STAGES(9);
      call = (unsigned)__builtin_amdgcn_readfirstlane((int)*(const LAS unsigned*)(lds + LDS_CTL));
      pollo = WS_STAGE + (call & 1u) * SLOT_BYTES + (((rank + 1u + (pl >> 2)) & 7u) << 20) + FLAG_OFF + (unsigned)pm * 64u + (pl & 3u) * 16u;
      pollo2 = lane == 63 ? WS_TMO : pollo;
      LOCAL_STAGES(1);
      POLL_ISSUE();
      LOCAL_STAGES(2);
      POLL_CHECK(2);
      LOCAL_STAGES(1);
      TO_STAGED(call);
      REMOTE_STAGES(9, ROLE);
      REMOTE_STAGES(2, 0);
      POLL_ISSUE();
      REMOTE_STAGES(2, 0);
      POLL_CHECK(16);
      if (gave_up || *(const LAS unsigned*)(lds + LDS_POLL + 63 * 16) == call) *(LAS unsigned*)(lds + LDS_CTL + 4) = 1u;
      REMOTE_STAGES(1, 0);
    } else {
      LOCAL_STAGES(13);
      TO_STAGED((unsigned)__builtin_amdgcn_readfirstlane((int)*(const LAS unsigned*)(lds + LDS_CTL)));
      if constexpr (ROLE != 0) { REMOTE_STAGES(9, ROLE); REMOTE_STAGES(5, 0); } else REMOTE_STAGES(14, 0);
    }
  };
  if (wid == 7) head(std::integral_constant<int, 7>{});
  else if (wid == 6) head(std::integral_constant<int, 6>{});
  else if (wid == 5) head(std::integral_constant<int, 5>{});
  else if (wid == 4) head(std::integral_constant<int, 4>{});
  else head(std::integral_constant<int, 0>{});

  wo = (((rank + 1u) & 7u) << 25) + (2u << 21);
  wslot = (30u % RING) * SLOTB; rslot = (27u % RING) * SLOTB;
#define TSM() ((void)0)
#define MAIN_STAGES(from, to, p) for (int s = (from); s < (to); ++s) { TSM(); wp = Wb + wo; ap = Ab + (wo >> 5); \
    READS(); ISSUE(0, APOL_R); NEXTK2(); ADV(); SCHED(); WAIT_V(8); WAIT_L0(); MMA_PHASE_P(p); }
  if (PACE_END > 27) MAIN_STAGES(27, PACE_END, PACE_M1)
  MAIN_STAGES(PACE_END, 125, PACE_M2)
  READS(); ADV(); SCHED(); WAIT_V(4); WAIT_L0(); MMA_PHASE_P(0);
  READS(); ADV(); SCHED(); WAIT_V(0); WAIT_L0(); MMA_PHASE_P(0);
  READS(); SCHED(); WAIT_L0(); MMA_PHASE_P(0);
  if (wr == 0) BAR();

  {
    const bool bad = __builtin_amdgcn_readfirstlane((int)*(const LAS unsigned*)(lds + LDS_CTL + 4)) != 0;
    const __amdgpu_buffer_rsrc_t orsrc = __builtin_amdgcn_make_buffer_rsrc(g.out, 0, MR * NN * 2, 0x00020000);
    LAS char* wl = lds + wid * 16384;
    const unsigned ooff = (unsigned)(((pm * 256 + wr * 128 + (lane >> 3)) * NN + pn * 256 + wc * 64 + (lane & 7) * 8) * 2);
    if (bad) {
      for (int i = 0; i < 16; ++i) __builtin_amdgcn_raw_buffer_store_b128((u32x4){0x7fc07fc0u, 0x7fc07fc0u, 0x7fc07fc0u, 0x7fc07fc0u}, orsrc, ooff + (unsigned)(i * 8) * (NN * 2), 0, 0);
    } else
#pragma unroll
    for (int m = 0; m < 8; ++m) {
#pragma unroll
      for (int np = 0; np < 2; ++np) {
        const f32x4 x = acc[m][2 * np], y = acc[m][2 * np + 1];
        const unsigned x0 = pk_bf16(silu(x[0]), silu(x[1])), x1 = pk_bf16(silu(x[2]), silu(x[3]));
        const unsigned y0 = pk_bf16(silu(y[0]), silu(y[1])), y1 = pk_bf16(silu(y[2]), silu(y[3]));
        const auto s0 = __builtin_amdgcn_permlane16_swap(x0, y0, false, false);
        const auto s1 = __builtin_amdgcn_permlane16_swap(x1, y1, false, false);
        const u32x4 w = {s0[0], s1[0], s0[1], s1[1]};
        const int c = np * 4 + (fq & 1) * 2 + (fq >> 1);
        *(LAS u32x4*)(wl + (m * 16 + fr) * 128 + ((c ^ (fr & 7)) * 16)) = w;
      }
#pragma unroll
      for (int h = 0; h < 2; ++h) {
        const int r = m * 16 + h * 8 + (lane >> 3);
        const u32x4 w = *(LAS u32x4*)(wl + r * 128 + (((lane & 7) ^ (r & 7)) * 16));
        __builtin_amdgcn_raw_buffer_store_b128(w, orsrc, ooff + (unsigned)(m * 16 + h * 8) * (NN * 2), 0, OAUX);
      }
    }
  }
}

}

extern "C" void kernel_launch(int rank, int nranks, ncclComm_t comm,
                              hipStream_t stream,
                              void* const* d_in, const int* in_sizes, int n_in,
                              void* d_out, int out_size,
                              const int* in_dtypes, int out_dtype,
                              void* const* d_out_all,
                              void* const* d_ws_all, size_t ws_bytes) {
  (void)comm; (void)in_dtypes; (void)out_dtype; (void)d_out_all;
  if (nranks != NRANK || (unsigned)rank >= (unsigned)NRANK || n_in != 2 || in_sizes[0] != NRANK * MR * KS || in_sizes[1] != NRANK * KS * NN ||
      out_size != MR * NN || ws_bytes < WS_NEED) {
    fprintf(stderr, "kernel_launch: unexpected shapes\n");
    return;
  }
  Params p;
  p.x = (const char*)d_in[0]; p.w = (const char*)d_in[1]; p.out = (char*)d_out; p.rank = rank; p.stamps = nullptr;
  for (int i = 0; i < NRANK; ++i) p.ws[i] = (char*)d_ws_all[i];
  hipLaunchKernelGGL(a2a_gemm_silu, dim3(NWG), dim3(512), 0, stream, p);
  HIP_CHECK(hipGetLastError());
}
```

```cpp
#include <hip/hip_runtime.h>
#include <rccl/rccl.h>
#include <cstdio>

#define HIP_CHECK(expr) do { hipError_t _e = (expr); if (_e != hipSuccess) { \
    fprintf(stderr, "HIP error %s at %s:%d\n", hipGetErrorString(_e), __FILE__, __LINE__); \
    return; } } while (0)

namespace {

#define LAS __attribute__((address_space(3)))
typedef short bf16x8 __attribute__((ext_vector_type(8)));
typedef short s16x4 __attribute__((ext_vector_type(4)));
typedef float f32x4 __attribute__((ext_vector_type(4)));
typedef unsigned u32x4 __attribute__((ext_vector_type(4)));
typedef unsigned u32x2 __attribute__((ext_vector_type(2)));
typedef float f32x2 __attribute__((ext_vector_type(2)));
typedef __bf16 bf16x2_t __attribute__((ext_vector_type(2)));

constexpr int NRANK = 8, MR = 512, KS = 512, NN = 32768;
constexpr int NWG = 256;
constexpr unsigned WS_TMO = 0, WS_DONE = 256;
constexpr unsigned WS_SYNC = 131072;
constexpr unsigned WS_CALLS = 65536;
constexpr unsigned WS_STAGE = 2u << 20, SLOT_BYTES = 8u << 20, FLAG_OFF = 32768;
constexpr size_t WS_NEED = (size_t)WS_STAGE + 2 * (size_t)SLOT_BYTES;
constexpr unsigned SLOTB = 32768;
constexpr unsigned LDS_POLL = 4 * SLOTB;
constexpr unsigned LDS_SYNC = LDS_POLL + 1024;
constexpr unsigned LDS_CTL = LDS_SYNC + 256;
constexpr unsigned LDS_BYTES = LDS_CTL + 32;
#define TS(i) ((void)0)
#define SPIN_COUNT() ((void)0)
#define PS(i) ((void)0)
#define WPOL " nt"
#define OAUX 19
#define PACE_T0 250u
#define PACE_DT 105u
#define PACE_DT2 70u
#define WKEEP 4
#define RECENT 25000u
#define KNEE 12
#define PUSH_FIRST 3
#define ORDER 0
#define PSTEP 1
#define SLOW_PAR 1u
#define SPACE_T0 PACE_T0
#define SPACE_DT 80u
#define SPACE_DT2 66u
#define CODE_WARM 1
#define END_WAIT 800u
#define EARLY_N 0u
#define LOCK 1
#define LOCK_DB 8u
#define LOCK_CAP 200
#define END_K 40u
constexpr unsigned long long DEADLINE_TICKS = 100000000ull;

struct Params { const char* xl; const char* w; const char* x; char* wsl; char* out; char* ws[NRANK]; int rank; unsigned long long* stamps; };

#define WAIT_V(n) asm volatile("s_waitcnt vmcnt(%0)" :: "n"(n) : "memory")
#define WAIT_L0() do { asm volatile("" ::: "memory"); __builtin_amdgcn_s_waitcnt(0xC07F); asm volatile("" ::: "memory"); } while (0)
#define BAR() do { asm volatile("" ::: "memory"); __builtin_amdgcn_s_barrier(); asm volatile("" ::: "memory"); } while (0)
#define SCHED() __builtin_amdgcn_sched_barrier(0)

__device__ __forceinline__ unsigned pk_bf16(f32x2 v) { bf16x2_t b = __builtin_convertvector(v, bf16x2_t); return __builtin_bit_cast(unsigned, b); }
typedef unsigned long long u64;
constexpr u64 ITEMS = 0x0eeeeeee01111111ull, DEADBIT = 1ull << 62;

__global__ __launch_bounds__(256, 1) void a2a_gemm_silu(Params g) {
  __shared__ __attribute__((aligned(1024))) char smem[LDS_BYTES];
  LAS char* lds = (LAS char*)smem;
  const int tid = threadIdx.x, wid = __builtin_amdgcn_readfirstlane(tid >> 6), lane = tid & 63;
  const int wr = wid >> 1, wc = wid & 1, fr = lane & 15, fq = lane >> 4;
  const int bid = blockIdx.x, pn = (bid & 7) * 16 + (bid >> 4), pm = (bid >> 3) & 1;
  const unsigned rank = (unsigned)g.rank & 7u;
  char* const wsl = g.wsl;
  const unsigned ldsbase = (unsigned)(size_t)lds;
  const unsigned tstart = (unsigned)__builtin_amdgcn_s_memrealtime();
  f32x4 acc[8][8];

  unsigned voffA[4], voffB[4];
  const int obp = lane * 16, ob = obp ^ (((obp >> 9) & 1) << 5);
#pragma unroll
  for (int i = 0; i < 4; ++i) voffA[i] = (unsigned)((pm * 256 + wid * 64 + i * 8 + (lane >> 3)) * 1024 + (((lane & 7) ^ (4 * (i & 1) + (lane >> 4))) * 16) + 4096 - 1024 * i);
  const unsigned m0l = ldsbase + (unsigned)((wid >> 1) * (int)SLOTB + (wid & 1) * 8192);
#pragma unroll
  for (int i = 0; i < 4; ++i) { const int kr = (wid * 4 + i) * 2 + (lane >> 5), f = (kr & 3) | (((kr >> 3) & 1) << 2), c = (lane & 31) ^ (f << 1);
    voffB[i] = (unsigned)(kr * 65536 + c * 16 + 4096 - 1024 * i); }
  const unsigned m0w = ldsbase + (unsigned)wid * 4096u;
  const char* const Wb = g.w + (size_t)pn * 512 - 4096;
  const char* ap = g.xl - 4096;
  const char* wp = Wb + ((size_t)rank << 25);
#define SETM0(v) asm volatile("s_mov_b32 m0, %0" :: "s"(v) : "memory")
#define GA(i) asm volatile("global_load_lds_dwordx4 %0, %1 offset:%2 sc0 sc1" :: "v"(voffA[i]), "s"(ap), "n"((i) * 1024) : "memory")
#define GB(i) asm volatile("global_load_lds_dwordx4 %0, %1 offset:%2" WPOL :: "v"(voffB[i]), "s"(wp), "n"((i) * 1024) : "memory")
#define GBK(i) do { if ((i) < WKEEP) asm volatile("global_load_lds_dwordx4 %0, %1 offset:%2" :: "v"(voffB[i]), "s"(wp), "n"((i) * 1024) : "memory"); else GB(i); } while (0)
#define ISSUE_B(slot) do { SETM0(m0w + (slot) * SLOTB + 16384u); asm volatile("s_nop 0"); GB(0); GB(1); GB(2); GB(3); wp += 1 << 21; } while (0)

  asm volatile("" : "+v"(voffA[0]), "+v"(voffA[1]), "+v"(voffA[2]), "+v"(voffA[3]), "+v"(voffB[0]), "+v"(voffB[1]), "+v"(voffB[2]), "+v"(voffB[3]));
  const unsigned dst = (rank + 1u + (unsigned)(bid >> 5)) & 7u, rg = (unsigned)bid & 31u;
  const char* const xs = g.x + (size_t)(dst * MR + rg * 16u) * 1024;
  const unsigned vx = (unsigned)((wid * 4 + ((lane >> 2) & 3)) * 1024 + fq * 64 + (lane & 3) * 16);
  u32x4 xq[4]; u32x2 cnt;
  PS(0);
  ISSUE_B(0);
  SETM0(m0l); asm volatile("s_nop 0"); GA(0); GA(1); GA(2); GA(3); ap += 32768;
  SETM0(m0l + 4096u); asm volatile("s_nop 0"); GA(0); GA(1); GA(2); GA(3); ap += 128 - 32768;
  asm volatile("global_load_dwordx4 %0, %2, %3 nt\n\tglobal_load_dwordx2 %1, %4, %5 sc1" : "=&v"(xq[0]), "=&v"(cnt) : "v"(vx), "s"(xs), "v"(WS_CALLS + (unsigned)bid * 128u), "s"(wsl) : "memory");
  PS(1);
  char* wsd = g.ws[dst]; char* outp = g.out;
  asm volatile("" : "+s"(wsd), "+s"(outp));

  unsigned ra, rb[8];
  unsigned ral = ldsbase + (unsigned)(wr * (int)SLOTB + fr * 128 + ((fq ^ (fr >> 1)) * 16));
  { const int o = fr * 64 + fq * 16; ra = ldsbase + (unsigned)(wr * 8192 + (o ^ (((o >> 9) & 1) << 5))); }
  { const int q = (lane & 15) >> 2, p = lane & 3, f = q | ((fq & 1) << 2);
#pragma unroll
    for (int n = 0; n < 8; ++n) rb[n] = ldsbase + (unsigned)(16384 + (8 * fq + q) * 512 + wc * 256 + ((n ^ f) * 32) + 8 * p); }
  asm volatile("" : "+v"(ral), "+v"(rb[0]), "+v"(rb[1]), "+v"(rb[2]), "+v"(rb[3]), "+v"(rb[4]), "+v"(rb[5]), "+v"(rb[6]), "+v"(rb[7]));
  bf16x8 A0[8], B0[8], A1[8], B1[8];
  s16x4 lo_;
#define RDA(dst_, m, so) dst_[m] = ((so) & 1u) ? *(const LAS bf16x8*)(size_t)((ral ^ (((so) & 65536u) | (((so) >> 9) & 64u))) + (m) * 2048) \
                                               : *(const LAS bf16x8*)(size_t)((ra ^ ((so) & 65536u)) + ((so) & 32768u) + (m) * 1024)
#define RDT(n, so, h) __builtin_amdgcn_ds_read_tr16_b64_v4i16((LAS s16x4*)(size_t)((rb[n] ^ ((so) & 65536u)) + ((so) & 32768u) + (h) * 2048))

#define MFMA(c, a, b) asm volatile("v_mfma_f32_16x16x32_bf16 %0, %1, %2, %0" : "+a"(c) : "v"(a), "v"(b))
#define MFMA0(c, a, b) asm volatile("v_mfma_f32_16x16x32_bf16 %0, %1, %2, 0" : "=a"(c) : "v"(a), "v"(b))
#define BUMP_A(d) do { ap += (d); asm volatile("" : "+s"(ap)); } while (0)
#define NEXT_A() BUMP_A(65536)
#define NEXT_W() do { wp += 1 << 21; asm volatile("" : "+s"(wp)); } while (0)
#define STAGE(Ac, Bc, An, Bn, so, ws, DOREAD, DOISSUE, GAP, CLOSE) do { \
    _Pragma("unroll") for (int m_ = 0; m_ < 8; ++m_) { \
      _Pragma("unroll") for (int n_ = 0; n_ < 8; ++n_) { const int gp = 8 * m_ + n_; \
        const int sn_ = (m_ & 1) ? 7 - n_ : n_, m = ORDER == 1 ? sn_ : ORDER == 2 ? (m_ & 6) + (((n_ + 1) >> 1) & 1) : m_, \
                  n = ORDER == 1 ? m_ : ORDER == 2 ? ((m_ & 2) ? 7 - ((m_ & 1) * 4 + (n_ >> 1)) : (m_ & 1) * 4 + (n_ >> 1)) : sn_; \
        if (DOREAD == 2) MFMA0(acc[m][n], Bc[n], Ac[m]); else MFMA(acc[m][n], Bc[n], Ac[m]); \
        SCHED(); \
        const int rd_ = DOREAD == 2 ? gp - 28 : DOREAD && (gp & 1) == 0 ? gp >> 1 : -1;     \
        if (rd_ >= 0 && rd_ < 16) { const int j_ = (rd_ >> 1) & 7; \
          if ((rd_ & 1) == 0) lo_ = RDT(j_, so, 0); \
          else { const s16x4 hi_ = RDT(j_, so, 1); Bn[j_] = __builtin_shufflevector(lo_, hi_, 0, 1, 2, 3, 4, 5, 6, 7); } } \
        if (rd_ >= 16 && rd_ < 24) RDA(An, (rd_ - 16) & 7, so); \
        if (DOISSUE & 1) { if (gp == 3) SETM0(m0w + (ws) * SLOTB); if (gp == 27) SETM0(m0w + (ws) * SLOTB + 16384u); \
          if (gp == 5 || gp == 11 || gp == 17 || gp == 23) GA((gp - 5) / 6); \
          if (gp == 29 || gp == 35 || gp == 41 || gp == 47) { if (DOISSUE & 8) GBK((gp - 29) / 6); else GB((gp - 29) / 6); } \
          if (!(DOISSUE & 4)) { if (gp == 49) NEXT_A(); if (gp == 51) NEXT_W(); } } \
        GAP; \
        SCHED(); } } \
    CLOSE; SCHED(); } while (0)
#define CLOSE_AT(vm) do { WAIT_V(vm); WAIT_L0(); BAR(); } while (0)

  PS(2); ISSUE_B(1); PS(3);
  {
    const char* pc; unsigned len;
    asm volatile("s_getpc_b64 %0\n.La2a_code0:\n\ts_mov_b32 %1, .La2a_code1-.La2a_code0" : "=s"(pc), "=s"(len));
    const unsigned co = min((unsigned)((((bid >> 3) * 4 + wid) & 63) * 1024 + lane * 16), (len - 16u) & ~15u);
    asm volatile("s_mov_b32 m0, %0\n\ts_nop 0\n\tglobal_load_lds_dwordx4 %1, %2" :: "s"(ldsbase + LDS_POLL), "v"(co), "s"((const char*)((size_t)pc & ~(size_t)15)) : "memory");
  }
  PS(4); WAIT_V(6 + CODE_WARM); PS(5); BAR(); PS(6);
#pragma unroll
  for (int n = 0; n < 8; ++n) { const s16x4 l_ = RDT(n, 0u, 0), h_ = RDT(n, 0u, 1); B0[n] = __builtin_shufflevector(l_, h_, 0, 1, 2, 3, 4, 5, 6, 7); RDA(A0, n, 1u); }
  WAIT_L0(); PS(7); SCHED();
  unsigned call, since; char* reg;
  const unsigned vpush = (unsigned)(fq * 65536 + (int)rg * 1024 + wid * 256 + fr * 16);
#define PUSH(k, mask) asm volatile("s_mov_b64 exec, %3\n\tglobal_store_dwordx4 %0, %1, %2 sc0 sc1 nt\n\ts_mov_b64 exec, -1\n\ts_nop 0" :: "v"(vpush), "v"(xq[k]), "s"(reg + (k) * 262144), "s"((u64)(mask)) : "memory")
#define REG_BASE (wsd + (rank << 20))
#define A2_GAP(pairslot, LOCAL) do { if (gp & 1) { const int q_ = gp >> 1; \
      if (q_ == 0) SETM0(LOCAL ? m0l + (pairslot) * SLOTB : m0w + (pairslot) * SLOTB); if (q_ >= 1 && q_ <= 4) GA((q_ - 1) & 3); \
      if (q_ == 5) { SETM0(LOCAL ? m0l + (pairslot) * SLOTB + 4096u : m0w + ((pairslot) + 1) * SLOTB); BUMP_A(LOCAL ? 32768 : 65536); } \
      if (q_ >= 6 && q_ <= 9) GA((q_ - 6) & 3); if (q_ == 10) BUMP_A(LOCAL ? 128 - 32768 : 65536); } } while (0)
#define B_GAP(g0, slot, STEP) do { if (gp == (g0)) SETM0(m0w + (slot) * SLOTB + 16384u); \
      if (gp > (g0) && gp <= (g0) + 8 && ((gp - (g0)) & 1) == 0) GB((((gp - (g0)) >> 1) - 1) & 3); if (STEP && gp == (g0) + 10) NEXT_W(); } while (0)
#define FIRST_GAP() do { A2_GAP(2, 1); B_GAP(29, 2, 1); B_GAP(41, 3, 1); \
    if (gp == 26) { asm volatile("s_waitcnt vmcnt(8)" : "+v"(xq[0]), "+v"(cnt) :: "memory"); BAR(); } \
    if (gp == 27) { call = (unsigned)__builtin_amdgcn_readfirstlane((int)cnt[0]) + 1u;     \
                    since = tstart - (unsigned)__builtin_amdgcn_readfirstlane((int)cnt[1]); asm volatile("" : "+s"(call), "+s"(since)); } \
    if (gp == 52) { reg = REG_BASE + WS_STAGE + (call & 1u) * SLOT_BYTES; asm volatile("" : "+s"(reg)); } \
    if (gp == 54) PUSH(0, 0xffffffffull); \
    if (gp == 56) asm volatile("global_load_dwordx4 %0, %3, %4 offset:256 nt\n\tglobal_load_dwordx4 %1, %3, %4 offset:512 nt\n\tglobal_load_dwordx4 %2, %3, %4 offset:768 nt" \
                               : "=&v"(xq[1]), "=&v"(xq[2]), "=&v"(xq[3]) : "v"(vx), "s"(xs) : "memory"); } while (0)
  TS(0);
  STAGE(A0, B0, A1, B1, 1 * SLOTB + 1, 0, 2, 0, FIRST_GAP(), CLOSE_AT(8));
  TS(1);
  STAGE(A1, B1, A0, B0, 2 * SLOTB + 1, 0, true, 0, A2_GAP(0, 1); B_GAP(29, 0, 1); B_GAP(41, 1, 1); if (gp == 54) PUSH(0, 0xffffffffull << 32), CLOSE_AT(21));
  TS(2);
  STAGE(A0, B0, A1, B1, 3 * SLOTB + 1, 0, true, 0, B_GAP(29, 2, 1),
        asm volatile("s_waitcnt vmcnt(9)" : "+v"(xq[1]), "+v"(xq[2]), "+v"(xq[3]) :: "memory"); WAIT_L0(); BAR());

  const char* const Ab = wsl + WS_STAGE + (call & 1u) * SLOT_BYTES - 4096;
  const unsigned pollo = lane == 63 ? WS_TMO : WS_STAGE + (call & 1u) * SLOT_BYTES + (((rank + 1u + (unsigned)(lane >> 2)) & 7u) << 20) + FLAG_OFF + (unsigned)(lane >> 5) * 4u;
#define STORE1(off, val, base, POL) asm volatile("s_mov_b64 exec, 1\n\tglobal_store_dword %0, %1, %2" POL "\n\ts_mov_b64 exec, -1" :: "v"(off), "v"(val), "s"(base) : "memory")
#define STORE2_W0(off, val, base) asm volatile("s_cmp_lg_u32 %3, 0\n\ts_cbranch_scc1 1f\n\ts_mov_b64 exec, 1\n\tglobal_store_dwordx2 %0, %1, %2 sc1\n\ts_mov_b64 exec, -1\n1:" \
    :: "v"(off), "v"(val), "s"(base), "s"(wid) : "memory", "scc")
#define ADD1_W0(off, base) asm volatile("s_cmp_lg_u32 %3, 0\n\ts_cbranch_scc1 1f\n\ts_mov_b64 exec, 1\n\tglobal_atomic_add %0, %1, %2 sc1\n\ts_mov_b64 exec, -1\n1:" \
    :: "v"(off), "v"(1u), "s"(base), "s"(wid) : "memory", "scc")
#define POLL_ISSUE() asm volatile("s_cmp_lg_u32 %3, 0\n\ts_cbranch_scc1 1f\n\ts_mov_b32 m0, %0\n\ts_nop 0\n\tglobal_load_lds_dword %1, %2 sc0 sc1\n1:" \
    :: "s"(ldsbase + LDS_POLL), "v"(pollo), "s"(wsl), "s"(nopoll) : "memory", "scc")
#define GRID_WAIT() do { const unsigned g_ = (unsigned)__builtin_amdgcn_s_memrealtime() >> 7; while (((unsigned)__builtin_amdgcn_s_memrealtime() >> 7) == g_) __builtin_amdgcn_s_sleep(1); } while (0)
  u64 l1, l2, avail = 0, done = 0, turn = ITEMS | DEADBIT, cand;
  unsigned ib, wo, nopoll = (unsigned)wid;
  unsigned pv;
  const unsigned need = ((call + 1u) >> 1) * 32u;
#define OHASH() ((void)0)
#define PINS(x) asm volatile("" : "+s"(x))
#define PINV(x) asm volatile("" : "+v"(x))
#define FLAG_STEP(k) do { \
    if (k == 0) { l1 = __builtin_amdgcn_ballot_w64(pv >= need); PINS(l1); } \
    if (k == 1) { avail |= l1 & ITEMS; PINS(avail); } \
    if (k == 2) { ib = (unsigned)__builtin_amdgcn_readlane((int)pv, 63); PINS(ib); } \
    if (k == 3) { avail = ib == call ? ITEMS | DEADBIT : avail; PINS(avail); } } while (0)
#define CHOOSE_STEP(k) do { \
    if (k == 0) { cand = avail & ~done; PINS(cand); } \
    if (k == 1) { l2 = turn & (turn - 1ull); PINS(l2); }            \
    if (k == 2) { l1 = cand & turn & ~l2; PINS(l1); }               \
    if (k == 3) { l1 = l1 ? l1 : cand & ~turn; PINS(l1); }          \
    if (k == 4) { ib = (unsigned)__builtin_ctzll(l1 | (1ull << 63)); PINS(ib); } \
    if (k == 5) { wo = 63u - (unsigned)__builtin_clzll((cand & ITEMS) | 2ull); PINS(wo); } \
    if (k == 6) { ib = l1 ? ib : wo; PINS(ib); }                    \
    if (k == 7) { done |= 1ull << ib; PINS(done); } \
    if (k == 8) { turn = cand ? l2 : turn; PINS(turn); } \
    if (k == 9) { wo = ((rank * 4u + 4u + ib) & 31u) << 23; PINS(wo); } \
    if (k == 10) { ap = Ab + (wo >> 5); PINS(ap); } \
    if (k == 11) { wp = Wb + wo; PINS(wp); OHASH(); } \
    if (k == 12) { l2 = turn & (turn - 1ull); PINS(l2); } \
    if (k == 13) { l1 = avail & ~done & ~l2; PINS(l1); } \
    if (k == 14) { nopoll = (unsigned)l1 | (unsigned)(l1 >> 32) | (unsigned)wid; PINS(nopoll); } } while (0)
#define EVAL_GAP() do { if (gp == 1) pv = *(const LAS unsigned*)(lds + LDS_POLL + lane * 4); \
    if (gp == 13) FLAG_STEP(0); if (gp == 15) FLAG_STEP(1); if (gp == 19) FLAG_STEP(2); if (gp == 21) FLAG_STEP(3); \
    if (gp == 25) CHOOSE_STEP(0); if (gp == 31) CHOOSE_STEP(1); if (gp == 33) CHOOSE_STEP(2); if (gp == 37) CHOOSE_STEP(3); if (gp == 39) CHOOSE_STEP(4); \
    if (gp == 43) CHOOSE_STEP(5); if (gp == 45) CHOOSE_STEP(6); if (gp >= 48 && (gp & 1) == 0) CHOOSE_STEP(7 + ((gp - 48) >> 1)); } while (0)
#define EVAL_END() do { if (cand == 0ull) { \
      if (wid == 0) { const u64 t0_ = __builtin_amdgcn_s_memrealtime(); \
        GRID_WAIT(); \
        for (;;) { SPIN_COUNT(); \
          asm volatile("global_load_dword %0, %1, %2 sc0 sc1\n\ts_waitcnt vmcnt(0)" : "=&v"(pv) : "v"(pollo), "s"(wsl) : "memory"); \
          _Pragma("unroll") for (int k_ = 0; k_ < 4; ++k_) FLAG_STEP(k_); \
          GRID_WAIT(); \
          if ((avail & ~done) != 0ull) break; \
          if (__builtin_amdgcn_s_memrealtime() - t0_ > DEADLINE_TICKS) { \
            STORE1(WS_TMO, call, wsl, " sc1"); avail = ITEMS | DEADBIT; break; } } \
        *(LAS u64*)(lds + LDS_CTL + 16) = avail; WAIT_L0(); } \
      BAR(); \
      const u64 mm_ = *(const LAS u64*)(lds + LDS_CTL + 16); \
      avail = ((u64)(unsigned)__builtin_amdgcn_readfirstlane((int)(mm_ >> 32)) << 32) | (unsigned)__builtin_amdgcn_readfirstlane((int)mm_); \
      _Pragma("unroll") for (int k_ = 0; k_ < 15; ++k_) CHOOSE_STEP(k_); } } while (0)
  unsigned now;
#define NOW_GAP() do { if (gp == 55) now = (unsigned)__builtin_amdgcn_s_memrealtime(); } while (0)
  unsigned xcc; asm volatile("s_getreg_b32 %0, hwreg(HW_REG_XCC_ID)" : "=s"(xcc));
  const bool slowx = (xcc & 1u) == SLOW_PAR;
#define DUE_(st, t0, d1, d2) ((t0) + ((st) <= KNEE ? (st) * (d1) : KNEE * (d1) + ((st) - KNEE) * (d2)))
#define PACE(st) PACE_TO(tstart + (slowx ? DUE_(st, SPACE_T0, SPACE_DT, SPACE_DT2) : DUE_(st, PACE_T0, PACE_DT, PACE_DT2)))
#define PACE_TO(d) do { const unsigned due_ = (d); while ((int)(now - due_) < 0) { __builtin_amdgcn_s_sleep(1); now = (unsigned)__builtin_amdgcn_s_memrealtime(); } } while (0)

  unsigned tj = tstart, tdur = 0, lag = 0, sj, st;
  u32x2 srec;
#define LOCK_POST(j) asm volatile("s_mov_b64 exec, 1\n\tglobal_store_dwordx2 %0, %1, %2 sc1\n\ts_mov_b64 exec, -1" \
    :: "v"(WS_SYNC + (unsigned)bid * 128u), "v"((u32x2){tj, call * 32u + (unsigned)(j)}), "s"(wsl) : "memory")
#define LOCK_FETCH() asm volatile("s_mov_b32 m0, %0\n\ts_nop 0\n\tglobal_load_lds_dword %1, %2 sc1" :: "s"(ldsbase + LDS_SYNC), "v"(vsync), "s"(wsl) : "memory")
  const unsigned vsync = WS_SYNC + (unsigned)(bid ^ 8) * 128u + (unsigned)(lane & 1) * 4u;
#define LOCK_GAP(j) do { if (gp == 43) srec = *(const LAS u32x2*)(lds + LDS_SYNC); \
    if (gp == 49) { st = (unsigned)__builtin_amdgcn_readfirstlane((int)srec[0]); PINS(st); } \
    if (gp == 51) { sj = call * 32u + (unsigned)(j) - (unsigned)__builtin_amdgcn_readfirstlane((int)srec[1]); PINS(sj); } \
    if (gp == 53) { lag = st + sj * tdur - tj - LOCK_DB; PINS(lag); } \
    if (gp == 55) { lag = (unsigned)max((int)lag, 0); PINS(lag); lag = (unsigned)min((int)lag, LOCK_CAP); PINS(lag); } \
    if (gp == 57) { lag = sj < 3u ? lag : 0u; PINS(lag); } \
    NOW_GAP(); } while (0)
#define LOCK_WAIT() do { tdur = now - tj; PACE_TO(now + lag); tj = now; } while (0)

#define UNIT_AT(S) (4 + (S - PUSH_FIRST) / PSTEP)
#define PUSH_LAST (PUSH_FIRST + 11 * PSTEP)
#define PUSHES(S) ((S) >= PUSH_FIRST && (S) <= PUSH_LAST && ((S) - PUSH_FIRST) % PSTEP == 0)
#define HEAD_GAP(S) do { \
    if (gp == 0 && S >= 9 && (S & 3) == 1) POLL_ISSUE(); \
    if (S < 16) { if (S & 1) A2_GAP((S + 3) & 3, S < 13); B_GAP(29, S & 3, !(S >= 11 && (S & 3) == 3)); } \
    if (gp == (S < 16 ? 41 : 7) && S >= PUSH_FIRST && S <= PUSH_LAST && (S - PUSH_FIRST) % PSTEP == 0) PUSH((UNIT_AT(S) >> 2) & 3, 0xffffull << (16 * (UNIT_AT(S) & 3))); \
    if (gp == 43 && (S == 4 || S == PUSH_LAST + 3)) ADD1_W0(FLAG_OFF + (S == 4 ? 0u : 4u), reg);     \
    if (gp == 43 && S == 6) STORE2_W0(WS_CALLS + (unsigned)bid * 128u, ((u32x2){call, tstart}), wsl); \
    if (S >= 11 && (S & 3) == 3) EVAL_GAP(); \
    if (S == 11 && gp == 25) { voffA[0] = voffA[1] = voffA[2] = voffA[3] = (unsigned)((pm * 256 + wid * 64) * 64 + ob + 4096); } \
    if (LOCK && S >= 16) { if ((S & 3) == 0 && gp == 13) LOCK_POST(S >> 2); if ((S & 3) == 1 && gp == 2) LOCK_FETCH(); if ((S & 3) == 3) LOCK_GAP(S >> 2); } \
    if (LOCK && (S == 11 || S == 15)) NOW_GAP(); \
    if (S >= 20 && (S & 3) == 0 && gp == 9) { nopoll = (avail & ITEMS) == ITEMS ? 1u + (unsigned)wid : (unsigned)wid; PINS(nopoll); } \
    if (PACED(S)) NOW_GAP(); } while (0)
#define PACED(S) (S < KNEE ? (S & 1) == 0 : S < 20 && (S & 3) == 2)
#define HEAD_CLOSE(S) do { if (PACED(S)) PACE(S + 1); if (LOCK && S >= 11 && (S & 3) == 3) LOCK_WAIT(); CLOSE_AT(S < 16 && (S & 1) == 0 ? 8 + PUSHES(S - 1) + PUSHES(S) : S == 16 ? 12 : 16); if (S >= 11 && (S & 3) == 3) EVAL_END(); } while (0)
#define HEAD0(S) do { TS(S); STAGE(A0, B0, A1, B1, ((S + 1) & 3) * SLOTB + (S < 15), (S & 3), true, (S < 16 ? 0 : (S & 3) == 3 ? 5 : 1), HEAD_GAP(S), HEAD_CLOSE(S)); } while (0)
#define HEAD1(S) do { TS(S); STAGE(A1, B1, A0, B0, ((S + 1) & 3) * SLOTB + (S < 15), (S & 3), true, (S < 16 ? 0 : (S & 3) == 3 ? 5 : 1), HEAD_GAP(S), HEAD_CLOSE(S)); } while (0)
  HEAD1(3); HEAD0(4); HEAD1(5); HEAD0(6); HEAD1(7); HEAD0(8); HEAD1(9); HEAD0(10); HEAD1(11);
  HEAD0(12); HEAD1(13); HEAD0(14); HEAD1(15); HEAD0(16); HEAD1(17); HEAD0(18); HEAD1(19);
  HEAD0(20); HEAD1(21); HEAD0(22); HEAD1(23); HEAD0(24); HEAD1(25); HEAD0(26); HEAD1(27);
  int j = 7;
#define SETTLED() ((avail & ITEMS) == ITEMS && ((done ^ turn) & ITEMS) == ITEMS)
  if (!SETTLED()) {
#pragma nounroll
    do {
      TS(4 * j);
      nopoll = (unsigned)wid;
      STAGE(A0, B0, A1, B1, 1 * SLOTB, 0, true, 1, (void)0, CLOSE_AT(16));
      STAGE(A1, B1, A0, B0, 2 * SLOTB, 1, true, 1, if (gp == 1) POLL_ISSUE(), CLOSE_AT(16));
      STAGE(A0, B0, A1, B1, 3 * SLOTB, 2, true, 1, (void)0, CLOSE_AT(16));
      STAGE(A1, B1, A0, B0, 0 * SLOTB, 3, true, 5, EVAL_GAP(), CLOSE_AT(16); EVAL_END());
      ++j;
    } while (!SETTLED());
  }
#define TURN_GAP() do { \
    if (gp == 48) { ib = (unsigned)__builtin_ctzll(turn); PINS(ib); } \
    if (gp == 49) { turn &= ~(1ull << ib); PINS(turn); } \
    if (gp == 51) { ib += rank * 4u + 4u; PINS(ib); } \
    if (gp == 52) { ib &= 31u; PINS(ib); } \
    if (gp == 53) { wo = ib << 18; PINS(wo); } \
    if (gp == 55) { ap = Ab + wo; PINS(ap); } \
    if (gp == 57) { wo = ib << 23; PINS(wo); OHASH(); } \
    if (gp == 59) { wp = Wb + wo; PINS(wp); } } while (0)
#define TRIPS(to, KEEP) _Pragma("nounroll") for (; j < (to); ++j) { \
    TS(4 * j); \
    STAGE(A0, B0, A1, B1, 1 * SLOTB, 0, true, (1 | KEEP), (void)0, CLOSE_AT(16)); \
    STAGE(A1, B1, A0, B0, 2 * SLOTB, 1, true, (1 | KEEP), (void)0, CLOSE_AT(16)); \
    STAGE(A0, B0, A1, B1, 3 * SLOTB, 2, true, (1 | KEEP), (void)0, CLOSE_AT(16)); \
    STAGE(A1, B1, A0, B0, 0 * SLOTB, 3, true, (5 | KEEP), TURN_GAP(), CLOSE_AT(16)); }
  const int jkeep = __builtin_amdgcn_readfirstlane((call > 1u && since < RECENT) ? 0 : 31);
  TRIPS(jkeep, 0)
  TRIPS(31, 8)
  STAGE(A0, B0, A1, B1, 1 * SLOTB, 0, true, 0, (void)0, CLOSE_AT(8));
  unsigned fin = 1u;
  unsigned wa[2], rda[2];
#pragma unroll
  for (int e = 0; e < 2; ++e) { wa[e] = ldsbase + (unsigned)(wid * 32768 + fr * 512 + (((4 * e + fq) ^ (fr & 7)) * 16));
                                rda[e] = ldsbase + (unsigned)(wid * 32768 + fq * 512 + (((2 * fr + e) ^ fq) * 16)); }
  unsigned ooff = (unsigned)(((pm * 256 + wr * 128 + fq) * NN + pn * 256 + wc * 128 + fr * 8) * 2);
  asm volatile("" : "+v"(wa[0]), "+v"(wa[1]), "+v"(rda[0]), "+v"(rda[1]), "+v"(ooff));
  bf16x8 A2[8], B2[8];
#define RD24(Ad, Bd, r, so) do { if ((r) >= 0 && (r) < 16) { if (((r) & 1) == 0) lo_ = RDT(((r) >> 1) & 7, so, 0); \
      else { const s16x4 hi_ = RDT(((r) >> 1) & 7, so, 1); Bd[((r) >> 1) & 7] = __builtin_shufflevector(lo_, hi_, 0, 1, 2, 3, 4, 5, 6, 7); } } \
    if ((r) >= 16 && (r) < 24) RDA(Ad, ((r) - 16) & 7, so); } while (0)
#define LAST_GAP() do { RD24(A0, B0, gp, 2 * SLOTB); \
    if (gp == 30) { WAIT_V(0); BAR(); }                             \
    if (gp == 31) asm volatile("s_cmp_lg_u32 %3, 0\n\ts_cbranch_scc1 1f\n\ts_mov_b64 exec, 1\n\tglobal_atomic_add %0, %1, %0, %2 sc0\n\ts_mov_b64 exec, -1\n1:" \
                               : "+v"(fin) : "v"(WS_DONE), "s"(wsl), "s"(wid) : "memory", "scc"); \
    RD24(A2, B2, gp - 32, 3 * SLOTB); } while (0)
  STAGE(A1, B1, A0, B0, 0u, 0, false, 0, LAST_GAP(), WAIT_L0(); BAR());
#define DUMP1(m, n) asm volatile("ds_write_b128 %0, %1 offset:%2" :: "v"(wa[(n) & 1]), "a"(acc[m][n]), "n"(((m) & 3) * 8192 + ((n) >> 1) * 128) : "memory")
#define TM(r, i) do { if ((r) < 8) { if ((i) < 8) MFMA(acc[(r) & 7][(i) & 7], B0[(i) & 7], A0[(r) & 7]); else MFMA(acc[(r) & 7][(15 - (i)) & 7], B2[(15 - (i)) & 7], A2[(r) & 7]); } } while (0)
#pragma unroll
  for (int i = 0; i < 16; ++i) TM(0, i);
#pragma unroll
  for (int i = 0; i < 16; ++i) { TM(1, i); if (i >= 2 && i < 10) DUMP1(0, (9 - i) & 7); }
  SCHED();

  {
    const __amdgpu_buffer_rsrc_t orsrc = __builtin_amdgcn_make_buffer_rsrc(outp, 0, MR * NN * 2, 0x00020000);
    if (avail & DEADBIT) {
      asm volatile("s_waitcnt vmcnt(0)" : "+v"(fin) :: "memory");
#pragma nounroll
      for (int i = 0; i < 32; ++i) __builtin_amdgcn_raw_buffer_store_b128((u32x4){0x7fc07fc0u, 0x7fc07fc0u, 0x7fc07fc0u, 0x7fc07fc0u}, orsrc, ooff + (unsigned)(i * 4) * (NN * 2), 0, 0);
    } else {
#define PIN16(a) do { asm volatile("" : "+v"(a[0]), "+v"(a[1]), "+v"(a[2]), "+v"(a[3]), "+v"(a[4]), "+v"(a[5]), "+v"(a[6]), "+v"(a[7]), \
                                        "+v"(a[8]), "+v"(a[9]), "+v"(a[10]), "+v"(a[11]), "+v"(a[12]), "+v"(a[13]), "+v"(a[14]), "+v"(a[15])); } while (0)
      f32x4 v[8];
      u32x4 wq[4];
#define RDBLK(b) do { _Pragma("unroll") for (int h = 0; h < 4; ++h) _Pragma("unroll") for (int e = 0; e < 2; ++e) \
        v[2 * h + e] = *(const LAS f32x4*)(size_t)((rda[e] ^ ((h & 1) * 64u)) + ((b) & 3) * 8192 + h * 2048); } while (0)
#define OUT(b, h) do { if ((b) >= 0) { SCHED(); __builtin_amdgcn_raw_buffer_store_b128(wq[h], orsrc, ooff + (unsigned)((b) * 16 + (h) * 4) * (NN * 2), 0, OAUX); SCHED(); } } while (0)
#define DUMP2(b, n) do { if ((b) < 7) { DUMP1(((b) + 1) & 7, n); DUMP1(((b) + 1) & 7, (n) + 1); } } while (0)
#define PIN4(a, o) asm volatile("" : "+v"(a[o]), "+v"(a[(o) + 1]), "+v"(a[(o) + 2]), "+v"(a[(o) + 3]))
#define PHASE(STMT, i0, K) do { _Pragma("unroll") for (int g_ = 0; g_ < 4; ++g_) { \
          _Pragma("unroll") for (int i = 4 * g_; i < 4 * g_ + 4; ++i) { STMT; } \
          PIN4(t, 4 * g_); if (((g_ + 1) * (K)) / 4 > (g_ * (K)) / 4) TM(b + 2, (i0) + (g_ * (K)) / 4); SCHED(); } } while (0)
      RDBLK(0);
#pragma unroll
      for (int b = 0; b < 8; ++b) {
        f32x2 y[16], t[16];
#pragma unroll
        for (int i = 0; i < 8; ++i) { y[2 * i] = (f32x2){v[i][0], v[i][1]}; y[2 * i + 1] = (f32x2){v[i][2], v[i][3]}; }
        if (b == 1 && END_WAIT) {
          if (wid == 0) {
            asm volatile("s_waitcnt vmcnt(0)" : "+v"(fin) :: "memory");
            const unsigned want = call * (unsigned)NWG - END_K, tw = (unsigned)__builtin_amdgcn_s_memrealtime();
            unsigned seen = (unsigned)__builtin_amdgcn_readfirstlane((int)fin) + 1u;
            if (seen - (call - 1u) * (unsigned)NWG <= EARLY_N) seen = want;
            while ((int)(seen - want) < 0 && (unsigned)__builtin_amdgcn_s_memrealtime() - tw < END_WAIT) {
              __builtin_amdgcn_s_sleep(8);
              seen = (unsigned)__builtin_amdgcn_readfirstlane((int)__hip_atomic_load((unsigned*)(wsl + WS_DONE), __ATOMIC_RELAXED, __HIP_MEMORY_SCOPE_AGENT));
            }
          }
          BAR();
        }
        OUT(b - 1, 0);
        PHASE(t[i] = y[i] * -1.44269504088896340736f, 0, 2);
        DUMP2(b, 6);
        PHASE((t[i][0] = __builtin_amdgcn_fmed3f(t[i][0], -86.5617024533378f, 86.5617024533378f), t[i][1] = __builtin_amdgcn_fmed3f(t[i][1], -86.5617024533378f, 86.5617024533378f)), 2, 4);
        OUT(b - 1, 1); DUMP2(b, 4);
        PHASE((t[i][0] = __builtin_amdgcn_exp2f(t[i][0]), t[i][1] = __builtin_amdgcn_exp2f(t[i][1])), 6, 4);
        OUT(b - 1, 2); DUMP2(b, 2);
        PHASE(t[i] = t[i] + 1.0f, 10, 2);
        DUMP2(b, 0);
        PHASE((t[i][0] = __builtin_amdgcn_rcpf(t[i][0]), t[i][1] = __builtin_amdgcn_rcpf(t[i][1])), 12, 4);
        OUT(b - 1, 3);
        PHASE(t[i] = y[i] * t[i], 16, 0);
        SCHED();
        if (b < 7) RDBLK(b + 1);
        SCHED();
#pragma unroll
        for (int h = 0; h < 4; ++h) wq[h] = (u32x4){pk_bf16(t[4 * h]), pk_bf16(t[4 * h + 1]), pk_bf16(t[4 * h + 2]), pk_bf16(t[4 * h + 3])};
      }
      OUT(7, 0); OUT(7, 1); OUT(7, 2); OUT(7, 3);
      asm volatile(".La2a_code1:");
    }
  }
}

}

extern "C" void kernel_launch(int rank, int nranks, ncclComm_t comm,
                              hipStream_t stream,
                              void* const* d_in, const int* in_sizes, int n_in,
                              void* d_out, int out_size,
                              const int* in_dtypes, int out_dtype,
                              void* const* d_out_all,
                              void* const* d_ws_all, size_t ws_bytes) {
  (void)comm; (void)in_dtypes; (void)out_dtype; (void)d_out_all;
  if (nranks != NRANK || (unsigned)rank >= (unsigned)NRANK || n_in != 2 || in_sizes[0] != NRANK * MR * KS || in_sizes[1] != NRANK * KS * NN ||
      out_size != MR * NN || ws_bytes < WS_NEED) {
    fprintf(stderr, "kernel_launch: unexpected shapes\n");
    return;
  }
  Params p;
  p.x = (const char*)d_in[0]; p.xl = p.x + (size_t)rank * MR * KS * 2; p.w = (const char*)d_in[1]; p.out = (char*)d_out; p.rank = rank; p.stamps = nullptr;
  for (int i = 0; i < NRANK; ++i) p.ws[i] = (char*)d_ws_all[i];
  p.wsl = p.ws[rank];
  hipLaunchKernelGGL(a2a_gemm_silu, dim3(NWG), dim3(256), 0, stream, p);
  HIP_CHECK(hipGetLastError());
}
```
